```python
import numpy as np

kernel_name = "gpu_ag_gemm_m4096_k4096_n4096_f32_reluqe4m3"


ABSMAX_RTOL = 0.09

COMPUTE_DTYPE = "float32"
EPILOGUE = "relu_quant_e4m3"
IN_QUANT = None
OUT_QUANT = "e4m3"
IN_DTYPES = {"x": "float32", "w_mat": "float32"}
WORLD_SIZE = 8

def setup_inputs():

    x = np.random.randn(4096, 4096).astype(np.float32)
    w_mat = (np.random.randn(4096, 4096) / np.sqrt(4096)).astype(np.float32)
    return {"x": x, "w_mat": w_mat}

def _snap_fp8(x, mant_bits, max_abs, min_normal):
    a = np.abs(x.astype(np.float64))
    _m, _e = np.frexp(a)
    q = np.where(a >= min_normal, np.ldexp(1.0, _e - 1 - mant_bits),
                 min_normal * 2.0 ** -mant_bits)
    snapped = np.minimum(np.round(a / q) * q, max_abs)
    return (np.sign(x) * snapped).astype(np.float32)
def _quant_dequant(y, amax):
    scale = amax / 448.0
    q = _snap_fp8(y / scale, 3, 448.0, 2.0 ** -6)
    return (q * scale).astype(np.float32)
def _epilogue(y):
    y = np.maximum(y, 0.0)
    return _quant_dequant(y, np.abs(y).max())

def reference(x, w_mat):
    y = (x @ w_mat).astype(np.float32)
    return _epilogue(y)
```

```cpp
#include <hip/hip_runtime.h>
#include <rccl/rccl.h>
#include <cstdio>
#include <cstdint>
#include <type_traits>

#define HIP_CHECK(expr) do { hipError_t _e = (expr); if (_e != hipSuccess) { \
    fprintf(stderr, "HIP error %s at %s:%d\n", hipGetErrorString(_e), __FILE__, __LINE__); \
    return; } } while (0)

namespace agk {

constexpr int NR = 8;
constexpr int KDIM = 4096, MS = 512, NS = 512;
constexpr int KC = 128;
constexpr int NCH = KDIM / KC;
constexpr int BK = 64;
constexpr int NST = KDIM / BK;
constexpr int SPC = KC / BK;
constexpr int AROWS = 1024, BCOLS = 2048;
constexpr int NXP = 32, NWP = 32, NCONS = 128;
constexpr int GRID = NXP + NWP + NCONS;

constexpr size_t A_OFF = 0;
constexpr size_t A_BYTES = (size_t)AROWS * KDIM * 2;
constexpr size_t B_OFF = A_OFF + A_BYTES;
constexpr size_t B_BYTES = (size_t)BCOLS * KDIM * 2;
constexpr size_t C_OFF = B_OFF + B_BYTES;
constexpr size_t XFLAG = C_OFF;
constexpr size_t WFLAG = C_OFF + 1024;
constexpr size_t DFLAG = C_OFF + 2048;
constexpr size_t CALLW = C_OFF + 4096;
constexpr size_t TMOW = C_OFF + 4096 + 128;
constexpr size_t GRAN = C_OFF + 8192;
constexpr size_t TRACE = C_OFF + 65536;
constexpr size_t WS_NEED = C_OFF + 65536 + 256 * 64 * 8;
constexpr unsigned long long DEADLINE_TICKS = 50000000ull;

constexpr int PF = 4;
constexpr int FD = 2;

typedef float f32x4 __attribute__((ext_vector_type(4)));
typedef float f32x2 __attribute__((ext_vector_type(2)));
typedef __bf16 bf16x2 __attribute__((ext_vector_type(2)));
typedef __bf16 bf16x8 __attribute__((ext_vector_type(8)));
typedef unsigned u32x4 __attribute__((ext_vector_type(4)));

struct Params {
    const float* x;
    const float* w;
    char* ws[NR];
    unsigned char* out[NR];
    int rank;
};

template <int N> __device__ __forceinline__ void wait_vm() {
    asm volatile("s_waitcnt vmcnt(%0)" ::"i"(N) : "memory");
}
template <int C, int END> struct Unroll {
    template <class F> static __device__ __forceinline__ void run(F& f) {
        f(std::integral_constant<int, C>{});
        Unroll<C + 1, END>::run(f);
    }
};
template <int END> struct Unroll<END, END> {
    template <class F> static __device__ __forceinline__ void run(F&) {}
};

__device__ __forceinline__ unsigned pk_bf16(float a, float b) {
    f32x2 v = {a, b};
    bf16x2 r = __builtin_convertvector(v, bf16x2);
    return __builtin_bit_cast(unsigned, r);
}
__device__ __forceinline__ unsigned ld_sys(const void* p) {
    return __hip_atomic_load((const unsigned*)p, __ATOMIC_RELAXED, __HIP_MEMORY_SCOPE_SYSTEM);
}
__device__ __forceinline__ unsigned ld_sys_asm(const void* p) {
    unsigned v;
    asm volatile("global_load_dword %0, %1, off sc0 sc1\n\ts_waitcnt vmcnt(0)" : "=&v"(v) : "v"(p) : "memory");
    return v;
}
__device__ __forceinline__ void st_sys(void* p, unsigned v) {
    __hip_atomic_store((unsigned*)p, v, __ATOMIC_RELAXED, __HIP_MEMORY_SCOPE_SYSTEM);
}
__device__ __forceinline__ __amdgpu_buffer_rsrc_t mk_rsrc(void* base, unsigned bytes) {
    return __builtin_amdgcn_make_buffer_rsrc(base, 0, bytes, 0x00020000);
}
#define STAMP(i) do {} while (0)

constexpr int prod_wait(int c, int S, int L) {
    int n = 0;
    for (int i = c - FD + 1; i <= c; ++i)
        if (i < NCH) n += S + ((i + PF < NCH) ? L : 0);
    return n;
}

__global__ void __launch_bounds__(256, 1) ag_gemm_kernel(Params p) {
    __shared__ __attribute__((aligned(16))) char smem[4 * 32768 + 256];
    const int rank = p.rank;
    char* me = p.ws[rank];
    const int tid = threadIdx.x, lane = tid & 63;
    const int wv = __builtin_amdgcn_readfirstlane(tid >> 6);
    const int bid = blockIdx.x;
    const int Q = rank >> 1;
    STAMP(0);
    const unsigned call = __builtin_amdgcn_readfirstlane(
        __hip_atomic_load((const unsigned*)(me + CALLW), __ATOMIC_RELAXED, __HIP_MEMORY_SCOPE_AGENT)) + 1u;
    const unsigned fbase = (call - 1u) * (unsigned)NCH;
    unsigned* tmo = (unsigned*)(me + TMOW);

    if (bid < NXP) {
        const int h = bid >> 2;
        int dq, g;
        if (h < 5) { dq = Q; g = 5 * (rank & 1) + h; }
        else { const int i = h - 5; dq = (Q + 1 + i) & 3; g = 10 + 2 * ((2 - i) & 3) + (rank & 1); }
        char* d0 = p.ws[(2 * dq) & 7];
        char* d1 = p.ws[(2 * dq + 1) & 7];
        const __amdgpu_buffer_rsrc_t r0 = mk_rsrc(d0 + A_OFF, (unsigned)A_BYTES);
        const __amdgpu_buffer_rsrc_t r1 = mk_rsrc(d1 + A_OFF, (unsigned)A_BYTES);
        const int row = bid * 16 + wv * 4 + (lane >> 4);
        const float* src = p.x + (size_t)row * KDIM + (lane & 15) * 8;
        const unsigned doff = (unsigned)((g * 64 + (row & 63)) * KC + (lane & 15) * 8) * 2u;
        unsigned* fl = (unsigned*)((lane == 0 ? d0 : d1) + XFLAG) + g * 4 + (bid & 3);
        f32x4 va[PF], vb[PF];
#pragma unroll
        for (int i = 0; i < PF; ++i) {
            va[i] = *(const f32x4*)(src + i * KC);
            vb[i] = *(const f32x4*)(src + i * KC + 4);
        }
        auto step = [&](auto CC) {
            constexpr int c = decltype(CC)::value;
            if constexpr (c < NCH) {
                constexpr int sl = c % PF;
                u32x4 v = {pk_bf16(va[sl].x, va[sl].y), pk_bf16(va[sl].z, va[sl].w),
                           pk_bf16(vb[sl].x, vb[sl].y), pk_bf16(vb[sl].z, vb[sl].w)};
                __builtin_amdgcn_raw_buffer_store_b128(v, r0, doff, c * (AROWS * KC * 2), 17);
                __builtin_amdgcn_raw_buffer_store_b128(v, r1, doff, c * (AROWS * KC * 2), 17);
                if constexpr (c + PF < NCH) {
                    va[sl] = *(const f32x4*)(src + (c + PF) * KC);
                    vb[sl] = *(const f32x4*)(src + (c + PF) * KC + 4);
                }
            }
            if constexpr (c >= FD) {
                wait_vm<prod_wait(c, 2, 2)>();
                __builtin_amdgcn_s_barrier();
                if (tid < 2) st_sys(fl, fbase + (unsigned)(c - FD + 1));
            }
        };
        Unroll<0, NCH + FD>::run(step);
        STAMP(1);
        if (bid != 0) return;
        {
            const int src_r = tid >> 4;
            const bool need = tid < 128 && (src_r >> 1) != Q;
            const unsigned* df = (const unsigned*)(me + DFLAG) + (tid & 127);
            if (wv < 2) {
                const unsigned long long t0 = __builtin_amdgcn_s_memrealtime();
                bool bad = false;
                while (true) {
                    const unsigned v = ld_sys(df);
                    if (__all(!need || v >= call)) break;
                    if (__hip_atomic_load(tmo, __ATOMIC_RELAXED, __HIP_MEMORY_SCOPE_AGENT) == call ||
                        __builtin_amdgcn_s_memrealtime() - t0 > DEADLINE_TICKS) { bad = true; break; }
                    __builtin_amdgcn_s_sleep(1);
                }
                if (bad && lane == 0) {
                    __hip_atomic_store(tmo, call, __ATOMIC_RELAXED, __HIP_MEMORY_SCOPE_AGENT);
                    p.out[rank][(size_t)(2 * Q) * MS * NS] = 0x7F;
                }
            }
            __syncthreads();
            if (tid == 0)
                __hip_atomic_store((unsigned*)(me + CALLW), call, __ATOMIC_RELAXED, __HIP_MEMORY_SCOPE_AGENT);
            STAMP(2);
        }
        return;
    }
    if (bid < NXP + NWP) {
        const int b = bid - NXP;
        const int g8 = b >> 2, kq = b & 3;
        const int hf = g8 >> 2;
        char* dd[4];
        unsigned cg[4];
        dd[0] = me; cg[0] = g8;
#pragma unroll
        for (int i = 0; i < 3; ++i) {
            const int dq = (Q + 1 + i) & 3;
            dd[i + 1] = p.ws[(2 * dq + hf) & 7];
            cg[i + 1] = (2 + 2 * ((2 - i) & 3) + (rank & 1)) * 4 + (g8 & 3);
        }
        __amdgpu_buffer_rsrc_t rs[4];
#pragma unroll
        for (int i = 0; i < 4; ++i) rs[i] = mk_rsrc(dd[i] + B_OFF + (size_t)cg[i] * 1024, (unsigned)(B_BYTES - cg[i] * 1024));
        const int col = ((lane >> 2) & 3) * 16 + (lane >> 4) * 4 + (lane & 3);
        const int k8w = kq * 4 + wv;
        const float* src = p.w + (size_t)(k8w * 8) * NS + g8 * 64 + col;
        const unsigned doff = (unsigned)(k8w * BCOLS + lane) * 16u;
        char* fd = lane == 0 ? dd[0] : lane == 1 ? dd[1] : lane == 2 ? dd[2] : dd[3];
        const unsigned fc = lane == 0 ? cg[0] : lane == 1 ? cg[1] : lane == 2 ? cg[2] : cg[3];
        unsigned* fl = (unsigned*)(fd + WFLAG) + fc * 4 + kq;
        float v[PF][8];
#pragma unroll
        for (int i = 0; i < PF; ++i)
#pragma unroll
            for (int j = 0; j < 8; ++j) v[i][j] = src[(size_t)(i * KC + j) * NS];
        auto step = [&](auto CC) {
            constexpr int c = decltype(CC)::value;
            if constexpr (c < NCH) {
                constexpr int sl = c % PF;
                u32x4 u = {pk_bf16(v[sl][0], v[sl][1]), pk_bf16(v[sl][2], v[sl][3]),
                           pk_bf16(v[sl][4], v[sl][5]), pk_bf16(v[sl][6], v[sl][7])};
#pragma unroll
                for (int i = 0; i < 4; ++i)
                    __builtin_amdgcn_raw_buffer_store_b128(u, rs[(i + 1) & 3], doff, c * ((KC / 8) * BCOLS * 16), 17);
                if constexpr (c + PF < NCH) {
#pragma unroll
                    for (int j = 0; j < 8; ++j) v[sl][j] = src[(size_t)((c + PF) * KC + j) * NS];
                }
            }
            if constexpr (c >= FD) {
                wait_vm<prod_wait(c, 4, 8)>();
                __builtin_amdgcn_s_barrier();
                if (tid < 4) st_sys(fl, fbase + (unsigned)(c - FD + 1));
            }
        };
        Unroll<0, NCH + FD>::run(step);
        STAMP(1);
        return;
    }

    const int cid = bid - (NXP + NWP);
    const int xg = cid & 7, xi = cid >> 3;
    const int rb = (xg & 1) * 4 + (xi & 3);
    const int cb = (xg >> 1) * 4 + (xi >> 2);
    const int wm = wv >> 1, wn = wv & 1;
    const int ml = lane & 15, q = lane >> 4;

    const char* xsrc = me + A_OFF + (size_t)(rb * 128 + wv * 32 + (lane >> 3)) * (KC * 2) + (((lane & 7) ^ (lane >> 3)) * 16);
    const char* wsrc = me + B_OFF + (size_t)((wv * 2) * BCOLS + cb * 128 + lane) * 16;
    auto issue = [&](int s) {
        char* xb = smem + (s & 3) * 32768 + wv * 4096;
        char* wb = smem + (s & 3) * 32768 + 16384 + wv * 4096;
        const char* xs = xsrc + (size_t)(s / SPC) * (AROWS * KC * 2) + (s % SPC) * (BK * 2);
        const char* wsn = wsrc + (size_t)s * (8 * BCOLS * 16);
#pragma unroll
        for (int i = 0; i < 4; ++i)
            __builtin_amdgcn_global_load_lds((const unsigned*)(xs + i * (8 * KC * 2)), (unsigned*)(xb + i * 1024), 16, 0, 17);
#pragma unroll
        for (int i = 0; i < 4; ++i)
            __builtin_amdgcn_global_load_lds((const unsigned*)(wsn + (i >> 1) * (BCOLS * 16) + (i & 1) * 1024), (unsigned*)(wb + i * 1024), 16, 0, 17);
    };
    const unsigned* pf = (lane & 8) ? (const unsigned*)(me + WFLAG) + cb * 8 + (lane & 7)
                                    : (const unsigned*)(me + XFLAG) + rb * 8 + (lane & 7);
    int ready = 0;
    bool bad = false;
    auto ensure = [&](int chunk) {
        if (wv == 0 && ready <= chunk && !bad) {
            const unsigned long long t0 = __builtin_amdgcn_s_memrealtime();
            while (true) {
                const unsigned v = ld_sys_asm(pf);
                while (ready < NCH && __all(v > fbase + (unsigned)ready)) ++ready;
                if (ready > chunk) break;
                if (__builtin_amdgcn_readfirstlane(ld_sys_asm(tmo)) == call ||
                    __builtin_amdgcn_s_memrealtime() - t0 > DEADLINE_TICKS) {
                    bad = true;
                    if (lane == 0) __hip_atomic_store(tmo, call, __ATOMIC_RELAXED, __HIP_MEMORY_SCOPE_AGENT);
                    break;
                }
                __builtin_amdgcn_s_sleep(1);
            }
        }
    };
    f32x4 acc[4][4];
#pragma unroll
    for (int u = 0; u < 4; ++u)
#pragma unroll
        for (int t = 0; t < 4; ++t) acc[u][t] = f32x4{0.f, 0.f, 0.f, 0.f};
    const int xrd = (wm * 64 + ml) * 128;
    const int wrd = 16384 + (q * 128 + wn * 64 + ml) * 16;
    auto compute = [&](int s) {
        const char* sb = smem + (s & 3) * 32768;
#pragma unroll
        for (int ks = 0; ks < 2; ++ks) {
            bf16x8 wf[4], xf[4];
#pragma unroll
            for (int t = 0; t < 4; ++t) wf[t] = *(const bf16x8*)(sb + wrd + ks * (4 * 128 * 16) + t * 256);
#pragma unroll
            for (int u = 0; u < 4; ++u) xf[u] = *(const bf16x8*)(sb + xrd + u * 2048 + (((ks * 4 + q) ^ (ml & 7)) * 16));
#pragma unroll
            for (int u = 0; u < 4; ++u)
#pragma unroll
                for (int t = 0; t < 4; ++t)
                    acc[u][t] = __builtin_amdgcn_mfma_f32_16x16x32_bf16(wf[t], xf[u], acc[u][t], 0, 0, 0);
        }
    };

    ensure(2 / SPC);
    __builtin_amdgcn_s_barrier();
    STAMP(1);
    issue(0); issue(1); issue(2);
    for (int s = 0; s < NST - 3; ++s) {
        if ((s + 3) % SPC == 0) ensure((s + 3) / SPC);
        wait_vm<16>();
        __builtin_amdgcn_s_barrier();
        issue(s + 3);
        compute(s);
    }
    STAMP(2);
    wait_vm<16>(); __builtin_amdgcn_s_barrier(); compute(NST - 3);
    wait_vm<8>();  __builtin_amdgcn_s_barrier(); compute(NST - 2);
    wait_vm<0>();  __builtin_amdgcn_s_barrier(); compute(NST - 1);
    STAMP(3);

    float mx = 0.f;
#pragma unroll
    for (int u = 0; u < 4; ++u)
#pragma unroll
        for (int t = 0; t < 4; ++t)
#pragma unroll
            for (int j = 0; j < 4; ++j) {
                acc[u][t][j] = fmaxf(acc[u][t][j], 0.f);
                mx = fmaxf(mx, acc[u][t][j]);
            }
#pragma unroll
    for (int o = 32; o > 0; o >>= 1) mx = fmaxf(mx, __shfl_xor(mx, o));
    float* red = (float*)(smem + 4 * 32768);
    if (lane == 0) { red[wv] = mx; if (wv == 0) red[8] = bad ? 1.f : 0.f; }
    __syncthreads();
    mx = fmaxf(fmaxf(red[0], red[1]), fmaxf(red[2], red[3]));
    bad = red[8] != 0.f;
    const size_t goff = GRAN + (size_t)(call & 1u) * (NR * NCONS * 8);
    if (tid < NR) {
        const unsigned long long gv = ((unsigned long long)call << 32) | __float_as_uint(mx);
        __hip_atomic_store((unsigned long long*)(p.ws[tid] + goff) + rank * NCONS + cid, gv, __ATOMIC_RELAXED, __HIP_MEMORY_SCOPE_SYSTEM);
    }
    {
        const unsigned long long* gp = (const unsigned long long*)(me + goff) + wv * 256 + lane;
        const unsigned long long t0 = __builtin_amdgcn_s_memrealtime();
        float am = 0.f;
        while (true) {
            unsigned long long g[4];
#pragma unroll
            for (int i = 0; i < 4; ++i) g[i] = __hip_atomic_load(gp + i * 64, __ATOMIC_RELAXED, __HIP_MEMORY_SCOPE_SYSTEM);
            bool ok = true;
            am = 0.f;
#pragma unroll
            for (int i = 0; i < 4; ++i) {
                ok = ok && (unsigned)(g[i] >> 32) == call;
                am = fmaxf(am, __uint_as_float((unsigned)g[i]));
            }
            if (__all(ok)) break;
            if (__hip_atomic_load(tmo, __ATOMIC_RELAXED, __HIP_MEMORY_SCOPE_AGENT) == call ||
                __builtin_amdgcn_s_memrealtime() - t0 > DEADLINE_TICKS) {
                bad = true;
                if (lane == 0) __hip_atomic_store(tmo, call, __ATOMIC_RELAXED, __HIP_MEMORY_SCOPE_AGENT);
                break;
            }
            __builtin_amdgcn_s_sleep(1);
        }
#pragma unroll
        for (int o = 32; o > 0; o >>= 1) am = fmaxf(am, __shfl_xor(am, o));
        if (lane == 0) { red[16 + wv] = am; red[24 + wv] = bad ? 1.f : 0.f; }
        __syncthreads();
        mx = fmaxf(fmaxf(red[16], red[17]), fmaxf(red[18], red[19]));
        bad = (red[24] + red[25] + red[26] + red[27]) != 0.f;
    }
    STAMP(4);
    {
        const float inv = mx > 0.f ? 448.0f / mx : 0.f;
        const int g = rb * 2 + wm;
        int srk, row0;
        if (g < 5) { srk = 2 * Q; row0 = 64 * g; }
        else if (g < 10) { srk = 2 * Q + 1; row0 = 64 * (g - 5); }
        else { const int j = g - 10, d = j >> 1; srk = 2 * ((Q + 1 + d) & 3) + (j & 1); row0 = 320 + 64 * ((2 - d) & 3); }
        const int slot = cb >> 1;
        int owner, coff;
        if (slot < 2) { owner = rank; coff = cb * 128; }
        else { const int j = slot - 2, d = j >> 1; owner = 2 * ((Q + 1 + d) & 3) + (j & 1); coff = 256 * (rank & 1) + (cb & 1) * 128; }
        owner &= 7;
        const __amdgpu_buffer_rsrc_t ro = mk_rsrc(p.out[owner], (unsigned)(KDIM * NS));
        const unsigned ooff = (unsigned)((srk * MS + row0 + ml) * NS + coff + wn * 64 + q * 16);
#pragma unroll
        for (int u = 0; u < 4; ++u) {
            u32x4 o;
#pragma unroll
            for (int t = 0; t < 4; ++t) {
                const float a0 = fminf(acc[u][t][0] * inv, 448.f), a1 = fminf(acc[u][t][1] * inv, 448.f);
                const float a2 = fminf(acc[u][t][2] * inv, 448.f), a3 = fminf(acc[u][t][3] * inv, 448.f);
                int r = __builtin_amdgcn_cvt_pk_fp8_f32(a0, a1, 0, false);
                r = __builtin_amdgcn_cvt_pk_fp8_f32(a2, a3, r, true);
                o[t] = bad ? 0x7F7F7F7Fu : (unsigned)r;
            }
            __builtin_amdgcn_raw_buffer_store_b128(o, ro, ooff, u * (16 * NS), 17);
        }
        wait_vm<0>();
        __syncthreads();
        if (tid == 0 && owner != rank)
            st_sys((unsigned*)(p.ws[owner] + DFLAG) + rank * 16 + rb * 2 + (cb & 1), call);
    }
    STAMP(5);
}

}

extern "C" void kernel_launch(int rank, int nranks, ncclComm_t comm,
                              hipStream_t stream,
                              void* const* d_in, const int* in_sizes, int n_in,
                              void* d_out, int out_size,
                              const int* in_dtypes, int out_dtype,
                              void* const* d_out_all,
                              void* const* d_ws_all, size_t ws_bytes) {
    (void)comm; (void)in_sizes; (void)n_in; (void)d_out; (void)out_size; (void)in_dtypes; (void)out_dtype;
    if (nranks != agk::NR || (unsigned)rank >= (unsigned)agk::NR || ws_bytes < agk::WS_NEED) {
        fprintf(stderr, "unsupported configuration\n");
        return;
    }
    agk::Params p;
    p.x = (const float*)d_in[0];
    p.w = (const float*)d_in[1];
    for (int i = 0; i < agk::NR; ++i) {
        p.ws[i] = (char*)d_ws_all[i];
        p.out[i] = (unsigned char*)d_out_all[i];
    }
    p.rank = rank;
    hipLaunchKernelGGL(agk::ag_gemm_kernel, dim3(agk::GRID), dim3(256), 0, stream, p);
    HIP_CHECK(hipGetLastError());
}
```

```cpp
#include <hip/hip_runtime.h>
#include <rccl/rccl.h>
#include <cstdio>
#include <cstdint>
#include <type_traits>

#define HIP_CHECK(expr) do { hipError_t _e = (expr); if (_e != hipSuccess) { \
    fprintf(stderr, "HIP error %s at %s:%d\n", hipGetErrorString(_e), __FILE__, __LINE__); \
    return; } } while (0)

namespace agk {

constexpr int NR = 8;
constexpr int KDIM = 4096, MS = 512, NS = 512;
constexpr int KC = 128;
constexpr int NCH = KDIM / KC;
constexpr int BK = 64;
constexpr int NST = KDIM / BK;
constexpr int SPC = KC / BK;
constexpr int AROWS = 1024, BCOLS = 2048;
constexpr int NXP = 56, NWP = 24, NCONS = 128;
constexpr int NSTREAM = 10, NITEMS = NCH * 8;
constexpr int GRID = NXP + NWP + NCONS + 1;

constexpr size_t A_OFF = 0;
constexpr size_t A_BYTES = (size_t)AROWS * KDIM * 2;
constexpr size_t B_OFF = A_OFF + A_BYTES;
constexpr size_t B_BYTES = (size_t)BCOLS * KDIM * 2;
constexpr size_t C_OFF = B_OFF + B_BYTES;
constexpr size_t DFLAG = C_OFF;
constexpr size_t LGRAN = C_OFF + 16384;
constexpr size_t GGRAN = C_OFF + 32768;
constexpr size_t TICKW = C_OFF + 36864;
constexpr size_t CALLW = C_OFF + 40960;
constexpr size_t TMOW = C_OFF + 40960 + 128;
constexpr size_t TRACE = C_OFF + 65536;
constexpr size_t IFLAG = C_OFF + 262144;
constexpr size_t WS_NEED = IFLAG + (size_t)NSTREAM * NITEMS * 128;
constexpr unsigned long long DEADLINE_TICKS = 50000000ull;

#define AGK_PF 2
#define AGK_LDAUX 17
#define DBG_NST NST
#define DBG_LDX(p) (*(const f32x4*)(p))
#define DBG_LDW(p) (*(p))
#define DBG_NL(n) (n)
#define DBG_KS 2
#define AGK_MAP 0
#define AGK_FD 1
constexpr int PF = AGK_PF;
constexpr int FD = AGK_FD;
constexpr unsigned long long SLOW_TICKS = 20000ull;

typedef float f32x4 __attribute__((ext_vector_type(4)));
typedef float f32x2 __attribute__((ext_vector_type(2)));
typedef __bf16 bf16x2 __attribute__((ext_vector_type(2)));
typedef __bf16 bf16x8 __attribute__((ext_vector_type(8)));
typedef unsigned u32x4 __attribute__((ext_vector_type(4)));

struct Params {
    const float* x;
    const float* w;
    char* ws[NR];
    unsigned char* out[NR];
    int rank;
};

template <int N> __device__ __forceinline__ void wait_vm() {
    asm volatile("s_waitcnt vmcnt(%0)" ::"i"(N) : "memory");
}
__device__ __forceinline__ void bar() { asm volatile("s_barrier" ::: "memory"); }
template <int C, int END> struct Unroll {
    template <class F> static __device__ __forceinline__ void run(F& f) {
        f(std::integral_constant<int, C>{});
        Unroll<C + 1, END>::run(f);
    }
};
template <int END> struct Unroll<END, END> {
    template <class F> static __device__ __forceinline__ void run(F&) {}
};

__device__ __forceinline__ unsigned pk_bf16(float a, float b) {
    f32x2 v = {a, b};
    bf16x2 r = __builtin_convertvector(v, bf16x2);
    return __builtin_bit_cast(unsigned, r);
}
__device__ __forceinline__ unsigned ld_sys(const void* p) {
    return __hip_atomic_load((const unsigned*)p, __ATOMIC_RELAXED, __HIP_MEMORY_SCOPE_SYSTEM);
}
__device__ __forceinline__ unsigned ld_sys_asm(const void* p) {
    unsigned v;
    asm volatile("global_load_dword %0, %1, off sc0 sc1\n\ts_waitcnt vmcnt(0)" : "=&v"(v) : "v"(p) : "memory");
    return v;
}
__device__ __forceinline__ int lds_ld(volatile int* p) {
    return __hip_atomic_load((int*)p, __ATOMIC_RELAXED, __HIP_MEMORY_SCOPE_WORKGROUP);
}
__device__ __forceinline__ int lds_ld_asm(unsigned lds_addr) {
    int v;
    asm volatile("ds_read_b32 %0, %1\n\ts_waitcnt lgkmcnt(0)" : "=&v"(v) : "v"(lds_addr) : "memory");
    return v;
}
__device__ __forceinline__ void lds_st(volatile int* p, int v) {
    __hip_atomic_store((int*)p, v, __ATOMIC_RELAXED, __HIP_MEMORY_SCOPE_WORKGROUP);
}
__device__ __forceinline__ void st_sys(void* p, unsigned v) {
    __hip_atomic_store((unsigned*)p, v, __ATOMIC_RELAXED, __HIP_MEMORY_SCOPE_SYSTEM);
}
__device__ __forceinline__ bool give_up(unsigned long long t0, unsigned* tmo, unsigned call) {
    const unsigned long long dt = __builtin_amdgcn_s_memrealtime() - t0;
    if (dt < SLOW_TICKS) return false;
    return dt > DEADLINE_TICKS || __builtin_amdgcn_readfirstlane(ld_sys_asm(tmo)) == call;
}
__device__ __forceinline__ __amdgpu_buffer_rsrc_t mk_rsrc(void* base, unsigned bytes) {
    return __builtin_amdgcn_make_buffer_rsrc(base, 0, bytes, 0x00020000);
}
#define STAMP(i) do {} while (0)
#define STAMPW(i, w) do {} while (0)
#define STAMPF(i) do {} while (0)

constexpr int prod_wait(int c, int S, int L) {
    int n = 0;
    for (int i = c - FD + 1; i <= c; ++i)
        if (i < NCH) n += S + ((i + PF < NCH) ? L : 0);
    return n;
}

__global__ void __launch_bounds__(512, 2) ag_gemm_kernel(Params p) {
    __shared__ __attribute__((aligned(16))) char smem[4 * 32768 + 256];
    const int rank = p.rank;
    char* me = p.ws[rank];
    const int tid = threadIdx.x, lane = tid & 63;
    const int wv = __builtin_amdgcn_readfirstlane(tid >> 6);
    const int bid = blockIdx.x;
    const int Q = rank >> 1;
    STAMP(0);
    const unsigned callv = __hip_atomic_load((const unsigned*)(me + CALLW), __ATOMIC_RELAXED, __HIP_MEMORY_SCOPE_AGENT);
    unsigned* tmo = (unsigned*)(me + TMOW);

    if (bid < NXP + NWP) {
        const int stream = bid >> 3, j0 = bid & 7;
        if (wv == 7 || (stream >= 8 && wv == 0)) return;
        volatile int* tk = (volatile int*)smem;
        const unsigned tk_a = (unsigned)(uintptr_t)(__attribute__((address_space(3))) char*)smem;
        int dst = rank, fidx = rank;
        bool isx = true;
        int xrow = 8 * wv, arow = 320 * (rank & 1) + 8 * wv, xrs = 40;
        int wcol = 0, slot = 0;
        if (stream < 6) {
            const int i = stream >> 1, m = stream & 1;
            const int sl = 2 * ((2 - i) & 3) + (rank & 1);
            dst = 2 * ((Q + 1 + i) & 3) + m;
            isx = wv == 0;
            xrow = 320 + 64 * i; arow = 640 + 64 * sl; xrs = 8;
            wcol = 256 * m; slot = 2 + sl;
        } else if (stream == 6) {
            dst = rank ^ 1;
        } else if (stream >= 8) {
            isx = false; fidx = stream; wcol = 256 * (stream - 8); slot = stream - 8;
        }
        char* dws = p.ws[dst & 7];
        if (wv == 6) {
            unsigned* tc = (unsigned*)(me + TICKW + stream * 128);
            int t1 = j0 + 8;
            for (int i = 0;; ++i) {
                int v = NITEMS;
                if (t1 < NITEMS) {
                    if (lane == 0) v = (int)__hip_atomic_fetch_add(tc, 1u, __ATOMIC_RELAXED, __HIP_MEMORY_SCOPE_AGENT) + 16;
                    v = __builtin_amdgcn_readfirstlane(v);
                    if (lane == 0) lds_st(tk + ((i + 2) & 7), v);
                    asm volatile("s_waitcnt lgkmcnt(0)" ::: "memory");
                }
                bar();
                if (t1 >= NITEMS) break;
                t1 = v;
            }
            bar();
            return;
        }
        if (wv == 5) {
            const unsigned call = __builtin_amdgcn_readfirstlane(callv) + 1u;
            char* fb = dws + IFLAG + (size_t)fidx * (NITEMS * 128);
            int tp = -1, t0 = j0, t1 = j0 + 8;
            for (int i = 0;; ++i) {
                bar();
                if (tp >= 0 && lane == 0) st_sys(fb + tp * 128, call);
                if (t1 >= NITEMS) break;
                tp = t0; t0 = t1;
                t1 = __builtin_amdgcn_readfirstlane(lds_ld_asm(tk_a + ((i + 2) & 7) * 4));
            }
            bar();
            if (lane == 0) st_sys(fb + t0 * 128, call);
            return;
        }
        if (isx) {
            const __amdgpu_buffer_rsrc_t r0 = mk_rsrc(dws + A_OFF, (unsigned)A_BYTES);
            const unsigned vo0 = (unsigned)((xrow + (lane >> 4)) * KDIM + (lane & 15) * 8) * 4u, vo1 = vo0 + 4 * KDIM * 4;
            const unsigned doff = (unsigned)((arow + (lane >> 4)) * KC + (lane & 15) * 8) * 2u;
            f32x4 va[4], vb[4];
            auto ld = [&](f32x4 (&v)[4], int t) {
                const float* sp = p.x + (size_t)((t & 7) * xrs) * KDIM + (t >> 3) * KC;
                asm volatile("s_nop 4\n\t"
                             "global_load_dwordx4 %0, %4, %6\n\tglobal_load_dwordx4 %1, %4, %6 offset:16\n\t"
                             "global_load_dwordx4 %2, %5, %6\n\tglobal_load_dwordx4 %3, %5, %6 offset:16"
                             : "=&v"(v[0]), "=&v"(v[1]), "=&v"(v[2]), "=&v"(v[3]) : "v"(vo0), "v"(vo1), "s"(sp) : "memory");
            };
            auto step = [&](f32x4 (&cur)[4], f32x4 (&nxt)[4], int t, int tn) {
                ld(nxt, tn < NITEMS ? tn : t);
                asm volatile("s_waitcnt vmcnt(6)" : "+v"(cur[0]), "+v"(cur[1]), "+v"(cur[2]), "+v"(cur[3]) :: "memory");
                const int so = ((t >> 3) * AROWS + (t & 7) * xrs) * (KC * 2);
#pragma unroll
                for (int e = 0; e < 2; ++e) {
                    const f32x4 a = cur[2 * e], b = cur[2 * e + 1];
                    u32x4 v = {pk_bf16(a.x, a.y), pk_bf16(a.z, a.w), pk_bf16(b.x, b.y), pk_bf16(b.z, b.w)};
                    __builtin_amdgcn_raw_buffer_store_b128(v, r0, doff + e * (4 * KC * 2), so, 17);
                }
                wait_vm<6>();
                bar();
            };
            int t = j0, tn = j0 + 8;
            ld(va, t);
            asm volatile("s_waitcnt vmcnt(0)" : "+v"(va[0]), "+v"(va[1]), "+v"(va[2]), "+v"(va[3]) :: "memory");
            for (int i = 0;; i += 2) {
                step(va, vb, t, tn);
                if (tn >= NITEMS) break;
                t = tn; tn = __builtin_amdgcn_readfirstlane(lds_ld_asm(tk_a + ((i + 2) & 7) * 4));
                step(vb, va, t, tn);
                if (tn >= NITEMS) break;
                t = tn; tn = __builtin_amdgcn_readfirstlane(lds_ld_asm(tk_a + ((i + 3) & 7) * 4));
            }
            asm volatile("s_waitcnt vmcnt(0)" : "+v"(va[0]), "+v"(va[1]), "+v"(va[2]), "+v"(va[3]),
                                                "+v"(vb[0]), "+v"(vb[1]), "+v"(vb[2]), "+v"(vb[3]) :: "memory");
        } else {
            const __amdgpu_buffer_rsrc_t r0 = mk_rsrc(dws + B_OFF, (unsigned)B_BYTES);
            const int col = ((lane >> 2) & 3) * 16 + (lane >> 4) * 4 + (lane & 3);
            const unsigned vo0 = (unsigned)(((wv - 1) * 16 + 2) * NS + wcol + col) * 4u;
            const unsigned vo1 = vo0 + 4 * NS * 4, vo2 = vo0 + 8 * NS * 4, vo3 = vo0 + 12 * NS * 4;
            const unsigned doff = (unsigned)((slot * (KC / 8) + (wv - 1) * 2) * 256 + lane) * 16u;
            float va[16], vb[16];
#define AGK_V16(c, v) c(v[0]), c(v[1]), c(v[2]), c(v[3]), c(v[4]), c(v[5]), c(v[6]), c(v[7]), \
                      c(v[8]), c(v[9]), c(v[10]), c(v[11]), c(v[12]), c(v[13]), c(v[14]), c(v[15])
            auto ld = [&](float (&v)[16], int t) {
                const float* sp = p.w + (size_t)((t >> 3) * KC + (t & 1) * 64) * NS + ((t & 7) >> 1) * 64;
                asm volatile("s_nop 4\n\t"
                             "global_load_dword %0, %16, %20 offset:-4096\n\tglobal_load_dword %1, %16, %20 offset:-2048\n\t"
                             "global_load_dword %2, %16, %20\n\tglobal_load_dword %3, %16, %20 offset:2048\n\t"
                             "global_load_dword %4, %17, %20 offset:-4096\n\tglobal_load_dword %5, %17, %20 offset:-2048\n\t"
                             "global_load_dword %6, %17, %20\n\tglobal_load_dword %7, %17, %20 offset:2048\n\t"
                             "global_load_dword %8, %18, %20 offset:-4096\n\tglobal_load_dword %9, %18, %20 offset:-2048\n\t"
                             "global_load_dword %10, %18, %20\n\tglobal_load_dword %11, %18, %20 offset:2048\n\t"
                             "global_load_dword %12, %19, %20 offset:-4096\n\tglobal_load_dword %13, %19, %20 offset:-2048\n\t"
                             "global_load_dword %14, %19, %20\n\tglobal_load_dword %15, %19, %20 offset:2048"
                             : AGK_V16("=&v", v) : "v"(vo0), "v"(vo1), "v"(vo2), "v"(vo3), "s"(sp) : "memory");
            };
            auto step = [&](float (&cur)[16], float (&nxt)[16], int t, int tn) {
                ld(nxt, tn < NITEMS ? tn : t);
                asm volatile("s_waitcnt vmcnt(18)" : AGK_V16("+v", cur) :: "memory");
                const int so = ((t >> 3) * ((KC / 8) * BCOLS) + (t & 1) * (8 * 256) + ((t & 7) >> 1) * 64) * 16;
#pragma unroll
                for (int e = 0; e < 2; ++e) {
                    u32x4 u = {pk_bf16(cur[8 * e], cur[8 * e + 1]), pk_bf16(cur[8 * e + 2], cur[8 * e + 3]),
                               pk_bf16(cur[8 * e + 4], cur[8 * e + 5]), pk_bf16(cur[8 * e + 6], cur[8 * e + 7])};
                    __builtin_amdgcn_raw_buffer_store_b128(u, r0, doff + e * (256 * 16), so, 17);
                }
                wait_vm<18>();
                bar();
            };
            int t = j0, tn = j0 + 8;
            ld(va, t);
            asm volatile("s_waitcnt vmcnt(0)" : AGK_V16("+v", va) :: "memory");
            for (int i = 0;; i += 2) {
                step(va, vb, t, tn);
                if (tn >= NITEMS) break;
                t = tn; tn = __builtin_amdgcn_readfirstlane(lds_ld_asm(tk_a + ((i + 2) & 7) * 4));
                step(vb, va, t, tn);
                if (tn >= NITEMS) break;
                t = tn; tn = __builtin_amdgcn_readfirstlane(lds_ld_asm(tk_a + ((i + 3) & 7) * 4));
            }
            asm volatile("s_waitcnt vmcnt(0)" : AGK_V16("+v", va) :: "memory");
            asm volatile("" : AGK_V16("+v", vb) :: "memory");
        }
        wait_vm<0>();
        bar();
        STAMPW(1, 1);
        return;
    }
    if (bid == GRID - 1) {
        if (wv >= 2) return;
        const unsigned call = __builtin_amdgcn_readfirstlane(callv) + 1u;
        volatile int* ctl = (volatile int*)(smem + 4 * 32768);
        char* gws = p.ws[0];
#pragma unroll
        for (int i = 1; i < NR; ++i) gws = (lane & 7) == i ? p.ws[i] : gws;
        unsigned long long* gdst = (unsigned long long*)(gws + GGRAN + ((call & 1u) * NR + rank) * 128);
        bool bad = false;
        {
            const unsigned long long* lg = (const unsigned long long*)(me + LGRAN + tid * 128);
            const unsigned long long t0 = __builtin_amdgcn_s_memrealtime();
            unsigned long long v;
            while (true) {
                v = __hip_atomic_load(lg, __ATOMIC_RELAXED, __HIP_MEMORY_SCOPE_AGENT);
                if (__all((unsigned)(v >> 32) == call)) break;
                if (give_up(t0, tmo, call)) { bad = true; break; }
                __builtin_amdgcn_s_sleep(2);
            }
            float am = __uint_as_float((unsigned)v);
#pragma unroll
            for (int o = 32; o > 0; o >>= 1) am = fmaxf(am, __shfl_xor(am, o));
            if (lane == 0) lds_st(ctl + wv, __float_as_int(am));
            __syncthreads();
            am = fmaxf(__int_as_float(lds_ld(ctl)), __int_as_float(lds_ld(ctl + 1)));
            if (tid < NR)
                __hip_atomic_store(gdst, ((unsigned long long)call << 32) | __float_as_uint(am), __ATOMIC_RELAXED, __HIP_MEMORY_SCOPE_SYSTEM);
        }
        STAMP(1);
        {
            const bool need = (tid >> 5) != Q;
            const unsigned* df = (const unsigned*)(me + DFLAG + tid * 128);
            const unsigned long long t0 = __builtin_amdgcn_s_memrealtime();
            while (true) {
                const unsigned v = ld_sys(df);
                if (__all(!need || v >= call)) break;
                if (give_up(t0, tmo, call)) { bad = true; break; }
                __builtin_amdgcn_s_sleep(2);
            }
            if (bad && lane == 0) {
                __hip_atomic_store(tmo, call, __ATOMIC_RELAXED, __HIP_MEMORY_SCOPE_AGENT);
                p.out[rank][(size_t)(2 * Q) * MS * NS] = 0x7F;
            }
            __syncthreads();
            if (tid == 0)
                __hip_atomic_store((unsigned*)(me + CALLW), call, __ATOMIC_RELAXED, __HIP_MEMORY_SCOPE_AGENT);
            if (tid < NSTREAM)
                __hip_atomic_store((unsigned*)(me + TICKW + tid * 128), 0u, __ATOMIC_RELAXED, __HIP_MEMORY_SCOPE_AGENT);
        }
        STAMP(2);
        return;
    }

    const int cid = bid - (NXP + NWP);
    const int xg = cid & 7, xi = cid >> 3;
    const int rb = (xg & 1) * 4 + (xi & 3);
    const int cb = (xg >> 1) * 4 + (xi >> 2);
    volatile int* ctl = (volatile int*)(smem + 4 * 32768);
    if (tid < 4) lds_st(ctl + tid, 0);
    __syncthreads();
    const unsigned call = __builtin_amdgcn_readfirstlane(callv) + 1u;

    if (wv >= 4) {
        const int lw = wv - 4;
        const char* xsrc = me + A_OFF + (size_t)(rb * 128 + lw * 32 + (lane >> 3)) * (KC * 2) + (((lane & 7) ^ (lane >> 3)) * 16);
        const char* wsrc = me + B_OFF + (size_t)(((cb >> 1) * (KC / 8) + lw * 2) * 256 + (cb & 1) * 128 + lane) * 16;
        auto issue = [&](int s) {
            char* xb = smem + (s & 3) * 32768 + lw * 4096;
            char* wb = smem + (s & 3) * 32768 + 16384 + lw * 4096;
            const char* xs = xsrc + (size_t)(s / SPC) * (AROWS * KC * 2) + (s % SPC) * (BK * 2);
            const char* wsn = wsrc + (size_t)(s / SPC) * ((KC / 8) * BCOLS * 16) + (s % SPC) * (8 * 256 * 16);
#pragma unroll
            for (int i = 0; i < 4; ++i)
                __builtin_amdgcn_global_load_lds((const unsigned*)(xs + i * (8 * KC * 2)), (unsigned*)(xb + i * 1024), 16, 0, AGK_LDAUX);
#pragma unroll
            for (int i = 0; i < 4; ++i)
                __builtin_amdgcn_global_load_lds((const unsigned*)(wsn + (i >> 1) * (256 * 16) + (i & 1) * 1024), (unsigned*)(wb + i * 1024), 16, 0, AGK_LDAUX);
        };
        const unsigned ctl_a = (unsigned)(uintptr_t)(__attribute__((address_space(3))) char*)(smem + 4 * 32768);
        int issued = 0;
        for (int s = 0; s < DBG_NST; ++s) {
            const int lim = s + 3 < NST ? s + 3 : NST;
            while (true) {
                const int rdy = __builtin_amdgcn_readfirstlane(lds_ld_asm(ctl_a)) * SPC;
                const int hi = rdy < lim ? rdy : lim;
                while (issued < hi) { issue(issued); ++issued; }
                if (issued > s) break;
                __builtin_amdgcn_s_sleep(1);
            }
            const int ahead = issued - s - 1;
            if (ahead == 0) wait_vm<0>(); else if (ahead == 1) wait_vm<8>(); else wait_vm<16>();
            bar();
        }
        if (lw != 0) return;
        const unsigned long long* gp = (const unsigned long long*)(me + GGRAN + ((call & 1u) * NR + (lane & 7)) * 128);
        const unsigned long long t0 = __builtin_amdgcn_s_memrealtime();
        unsigned long long v;
        bool bad = false;
        while (true) {
            v = __hip_atomic_load(gp, __ATOMIC_RELAXED, __HIP_MEMORY_SCOPE_SYSTEM);
            if (__all((unsigned)(v >> 32) == call)) break;
            if (give_up(t0, tmo, call)) {
                bad = true;
                if (lane == 0) __hip_atomic_store(tmo, call, __ATOMIC_RELAXED, __HIP_MEMORY_SCOPE_AGENT);
                break;
            }
            __builtin_amdgcn_s_sleep(2);
        }
        float am = __uint_as_float((unsigned)v);
#pragma unroll
        for (int o = 4; o > 0; o >>= 1) am = fmaxf(am, __shfl_xor(am, o));
        if (lane == 0) { lds_st(ctl + 8, __float_as_int(am)); if (bad) lds_st(ctl + 1, 1); }
        asm volatile("s_waitcnt lgkmcnt(0)" ::: "memory");
        bar();
        return;
    }

    const int wm = wv >> 1, wn = wv & 1;
    const int ml = lane & 15, q = lane >> 4;
    auto grp_src = [&](int g) { return g < 5 ? 2 * Q : g < 10 ? 2 * Q + 1 : 2 * ((Q + 1 + ((g - 10) >> 1)) & 3) + ((g - 10) & 1); };
    const int csrc = (cb >> 1) < 2 ? 8 + (cb >> 1) : 2 * ((Q + 1 + (((cb >> 1) - 2) >> 1)) & 3) + (((cb >> 1) - 2) & 1);
    const int psel = (lane >> 3) & 3;
    const int psrc = psel == 0 ? grp_src(2 * rb) : psel == 1 ? grp_src(2 * rb + 1) : csrc;
    const char* pfb = me + IFLAG + (size_t)psrc * (NITEMS * 128) + (lane & 7) * 128;
    int ready = 0;
    auto pf_at = [&](int r) { const int c = r + (lane >> 5); return pfb + (c < NCH ? c : NCH - 1) * 1024; };
    auto advance = [&](unsigned v, int r0) {
        const unsigned long long ok = __ballot(v == call);
        if (ready == r0 && (unsigned)ok == 0xFFFFFFFFu) {
            ++ready;
            if (ready < NCH && (unsigned)(ok >> 32) == 0xFFFFFFFFu) ++ready;
            if (lane == 0) lds_st(ctl, ready);
        }
    };
    auto poll = [&](int need) {
        const unsigned long long t0 = __builtin_amdgcn_s_memrealtime();
        while (true) {
            const int r0 = ready;
            advance(ld_sys_asm(pf_at(r0)), r0);
            if (ready > need) break;
            if (ready != r0) continue;
            if (give_up(t0, tmo, call)) {
                ready = NCH;
                if (lane == 0) {
                    __hip_atomic_store(tmo, call, __ATOMIC_RELAXED, __HIP_MEMORY_SCOPE_AGENT);
                    lds_st(ctl + 1, 1);
                    lds_st(ctl, ready);
                }
                break;
            }
            __builtin_amdgcn_s_sleep(1);
        }
    };
    f32x4 acc[4][4];
#pragma unroll
    for (int u = 0; u < 4; ++u)
#pragma unroll
        for (int t = 0; t < 4; ++t) acc[u][t] = f32x4{0.f, 0.f, 0.f, 0.f};
    const int xrd = (wm * 64 + ml) * 128;
    const int wrd = 16384 + (q * 128 + wn * 64 + ml) * 16;
    unsigned pv = 0;
    if (wv == 0 && ready < NCH) {
        poll(0);
        if (ready < NCH) asm volatile("global_load_dword %0, %1, off sc0 sc1" : "=v"(pv) : "v"(pf_at(ready)) : "memory");
    }
    STAMP(1);
    bf16x8 wf0[4], xf0[4], wf1[4], xf1[4];
#pragma unroll
    for (int t = 0; t < 4; ++t) { wf1[t] = bf16x8{}; xf1[t] = bf16x8{}; }
    const int xsw0 = (q ^ (ml & 7)) * 16, xsw1 = ((4 + q) ^ (ml & 7)) * 16;
    for (int s = 0; s < DBG_NST; ++s) {
        bar();
        if ((s & 1) == 0) STAMPF(8 + (s >> 1));
        const char* sb = smem + (s & 3) * 32768;
#pragma unroll
        for (int t = 0; t < 4; ++t) wf0[t] = *(const bf16x8*)(sb + wrd + t * 256);
#pragma unroll
        for (int u = 0; u < 4; ++u) xf0[u] = *(const bf16x8*)(sb + xrd + u * 2048 + xsw0);
        __builtin_amdgcn_sched_barrier(0);
#pragma unroll
        for (int u = 0; u < 4; ++u)
#pragma unroll
            for (int t = 0; t < 4; ++t)
                acc[u][t] = __builtin_amdgcn_mfma_f32_16x16x32_bf16(wf1[t], xf1[u], acc[u][t], 0, 0, 0);
        __builtin_amdgcn_sched_barrier(0);
#pragma unroll
        for (int t = 0; t < 4; ++t) wf1[t] = *(const bf16x8*)(sb + wrd + (4 * 128 * 16) + t * 256);
#pragma unroll
        for (int u = 0; u < 4; ++u) xf1[u] = *(const bf16x8*)(sb + xrd + u * 2048 + xsw1);
        __builtin_amdgcn_sched_barrier(0);
#pragma unroll
        for (int u = 0; u < 4; ++u)
#pragma unroll
            for (int t = 0; t < 4; ++t)
                acc[u][t] = __builtin_amdgcn_mfma_f32_16x16x32_bf16(wf0[t], xf0[u], acc[u][t], 0, 0, 0);
        __builtin_amdgcn_sched_barrier(0);
        asm volatile("s_waitcnt lgkmcnt(0)" ::: "memory");
        if (wv == 0 && ready < NCH) {
            asm volatile("s_waitcnt vmcnt(0)" : "+v"(pv) :: "memory");
            advance(pv, ready);
            if (ready <= (s + 1) / SPC) poll((s + 1) / SPC);
            if (ready < NCH) asm volatile("global_load_dword %0, %1, off sc0 sc1" : "=v"(pv) : "v"(pf_at(ready)) : "memory");
        }
    }
#pragma unroll
    for (int u = 0; u < 4; ++u)
#pragma unroll
        for (int t = 0; t < 4; ++t)
            acc[u][t] = __builtin_amdgcn_mfma_f32_16x16x32_bf16(wf1[t], xf1[u], acc[u][t], 0, 0, 0);
    STAMP(3);

    float mx = 0.f;
#pragma unroll
    for (int u = 0; u < 4; ++u)
#pragma unroll
        for (int t = 0; t < 4; ++t)
#pragma unroll
            for (int j = 0; j < 4; ++j) {
                acc[u][t][j] = fmaxf(acc[u][t][j], 0.f);
                mx = fmaxf(mx, acc[u][t][j]);
            }
#pragma unroll
    for (int o = 32; o > 0; o >>= 1) mx = fmaxf(mx, __shfl_xor(mx, o));
    {
        int tk = 0;
        if (lane == 0) {
            __hip_atomic_fetch_max((unsigned*)(ctl + 2), __float_as_uint(mx), __ATOMIC_RELAXED, __HIP_MEMORY_SCOPE_WORKGROUP);
            tk = __hip_atomic_fetch_add((int*)(ctl + 3), 1, __ATOMIC_RELAXED, __HIP_MEMORY_SCOPE_WORKGROUP);
        }
        tk = __builtin_amdgcn_readfirstlane(tk);
        if (tk == 3) {
            const unsigned tm = (unsigned)lds_ld(ctl + 2);
            if (lane == 0)
                __hip_atomic_store((unsigned long long*)(me + LGRAN + cid * 128), ((unsigned long long)call << 32) | tm,
                                   __ATOMIC_RELAXED, __HIP_MEMORY_SCOPE_AGENT);
        }
    }
    const int g = rb * 2 + wm;
    int srk, row0;
    if (g < 5) { srk = 2 * Q; row0 = 64 * g; }
    else if (g < 10) { srk = 2 * Q + 1; row0 = 64 * (g - 5); }
    else { const int j = g - 10, d = j >> 1; srk = 2 * ((Q + 1 + d) & 3) + (j & 1); row0 = 320 + 64 * ((2 - d) & 3); }
    const int slot = cb >> 1;
    int owner, coff;
    if (slot < 2) { owner = rank; coff = cb * 128; }
    else { const int j = slot - 2, d = j >> 1; owner = 2 * ((Q + 1 + d) & 3) + (j & 1); coff = 256 * (rank & 1) + (cb & 1) * 128; }
    owner &= 7;
    const __amdgpu_buffer_rsrc_t ro = mk_rsrc(p.out[owner], (unsigned)(KDIM * NS));
    const unsigned ooff = (unsigned)((srk * MS + row0 + ml) * NS + coff + wn * 64 + q * 16);
    unsigned* dfl = (unsigned*)(p.ws[owner] + DFLAG + (rank * 16 + rb * 2 + (cb & 1)) * 128);
    bar();
    STAMP(4);
    mx = __int_as_float(lds_ld(ctl + 8));
    const bool bad = lds_ld(ctl + 1) != 0;
    const float inv = mx > 0.f ? 448.0f / mx : 0.f;
#pragma unroll
    for (int u = 0; u < 4; ++u) {
        u32x4 o;
#pragma unroll
        for (int t = 0; t < 4; ++t) {
            const float a0 = fminf(acc[u][t][0] * inv, 448.f), a1 = fminf(acc[u][t][1] * inv, 448.f);
            const float a2 = fminf(acc[u][t][2] * inv, 448.f), a3 = fminf(acc[u][t][3] * inv, 448.f);
            int r = __builtin_amdgcn_cvt_pk_fp8_f32(a0, a1, 0, false);
            r = __builtin_amdgcn_cvt_pk_fp8_f32(a2, a3, r, true);
            o[t] = bad ? 0x7F7F7F7Fu : (unsigned)r;
        }
        __builtin_amdgcn_raw_buffer_store_b128(o, ro, ooff, u * (16 * NS), 17);
    }
    wait_vm<0>();
    bar();
    if (tid == 0 && owner != rank) st_sys(dfl, call);
    STAMP(5);
}

}

extern "C" void kernel_launch(int rank, int nranks, ncclComm_t comm,
                              hipStream_t stream,
                              void* const* d_in, const int* in_sizes, int n_in,
                              void* d_out, int out_size,
                              const int* in_dtypes, int out_dtype,
                              void* const* d_out_all,
                              void* const* d_ws_all, size_t ws_bytes) {
    (void)comm; (void)in_sizes; (void)n_in; (void)d_out; (void)out_size; (void)in_dtypes; (void)out_dtype;
    if (nranks != agk::NR || (unsigned)rank >= (unsigned)agk::NR || ws_bytes < agk::WS_NEED) {
        fprintf(stderr, "unsupported configuration\n");
        return;
    }
    agk::Params p;
    p.x = (const float*)d_in[0];
    p.w = (const float*)d_in[1];
    for (int i = 0; i < agk::NR; ++i) {
        p.ws[i] = (char*)d_ws_all[i];
        p.out[i] = (unsigned char*)d_out_all[i];
    }
    p.rank = rank;
    hipLaunchKernelGGL(agk::ag_gemm_kernel, dim3(agk::GRID), dim3(512), 0, stream, p);
    HIP_CHECK(hipGetLastError());
}
```

```cpp
#include <hip/hip_runtime.h>
#include <rccl/rccl.h>
#include <cstdio>
#include <cstdint>

#define HIP_CHECK(expr) do { hipError_t _e = (expr); if (_e != hipSuccess) { \
    fprintf(stderr, "HIP error %s at %s:%d\n", hipGetErrorString(_e), __FILE__, __LINE__); \
    return; } } while (0)

namespace agk {

constexpr int NR = 8;
constexpr int KDIM = 4096, MS = 512, NS = 512;
constexpr int BK = 64;
constexpr int NST = KDIM / BK;
constexpr int AROWS = 1024, BCOLS = 2048;
constexpr int NSTREAM = 10;
constexpr int NSL = 4;
constexpr int NITEMS = NST * NSL;
constexpr int NCONS = 128;
#define AGK_PWN 2
constexpr int PWN = AGK_PWN;
#define AGK_HG 48
#define AGK_HM 0xFFFFull
#define AGK_NRAW 4
#define AGK_NRAW0 0
constexpr int NPL0 = AGK_NRAW0, NPL = NST - AGK_NRAW;
constexpr int NTHR = 576;
constexpr int RWG = 5;
constexpr int PB0 = 8, CB0 = PB0 + NSTREAM * 8, RB0 = CB0 + NCONS, GRID = RB0 + 7 * RWG;

constexpr size_t A_OFF = 0;
constexpr size_t A_BYTES = (size_t)AROWS * KDIM * 2;
constexpr size_t B_OFF = A_OFF + A_BYTES;
constexpr size_t B_BYTES = (size_t)BCOLS * KDIM * 2;
constexpr int BPS = 192;
constexpr size_t D_OFF = B_OFF + B_BYTES;
constexpr size_t D_BYTES = (size_t)NST * BPS * 1536;
constexpr size_t H_OFF = D_OFF + D_BYTES;
constexpr size_t H_BYTES = (size_t)NST * BPS * 512;
constexpr size_t C_OFF = H_OFF + 2 * H_BYTES;
constexpr size_t DFLAG = C_OFF;
constexpr size_t LGRAN = C_OFF + 16384;
constexpr size_t GGRAN = C_OFF + 32768;
constexpr size_t TICKW = C_OFF + 36864;
constexpr size_t CALLW = C_OFF + 40960;
constexpr size_t TMOW = C_OFF + 40960 + 128;
constexpr size_t ZFLAG = C_OFF + 45056;
constexpr size_t TRACE = C_OFF + 65536;
constexpr size_t IFLAG = C_OFF + 262144;
constexpr size_t PFLAG = IFLAG + (size_t)NSTREAM * NITEMS * 128;
constexpr size_t WS_NEED = PFLAG + (size_t)NR * NITEMS * 128;
constexpr unsigned long long DEADLINE_TICKS = 50000000ull;
constexpr unsigned long long SLOW_TICKS = 20000ull;

typedef float f32x4 __attribute__((ext_vector_type(4)));
typedef float f32x2 __attribute__((ext_vector_type(2)));
typedef __bf16 bf16x2 __attribute__((ext_vector_type(2)));
typedef __bf16 bf16x8 __attribute__((ext_vector_type(8)));
typedef unsigned u32x4 __attribute__((ext_vector_type(4)));

struct Params {
    const float* x;
    const float* w;
    char* ws[NR];
    unsigned char* out[NR];
    int rank;
};

template <int N> __device__ __forceinline__ void wait_vm() {
    asm volatile("s_waitcnt vmcnt(%0)" ::"i"(N) : "memory");
}
__device__ __forceinline__ void bar() { asm volatile("s_barrier" ::: "memory"); }

__device__ __forceinline__ unsigned pk_bf16(float a, float b) {
    f32x2 v = {a, b};
    bf16x2 r = __builtin_convertvector(v, bf16x2);
    return __builtin_bit_cast(unsigned, r);
}
__device__ __forceinline__ unsigned ld_sys_asm(const void* p) {
    unsigned v;
    asm volatile("global_load_dword %0, %1, off sc0 sc1\n\ts_waitcnt vmcnt(0)" : "=&v"(v) : "v"(p) : "memory");
    return v;
}
__device__ __forceinline__ int lds_ld(volatile int* p) {
    return __hip_atomic_load((int*)p, __ATOMIC_RELAXED, __HIP_MEMORY_SCOPE_WORKGROUP);
}
__device__ __forceinline__ int lds_ld_asm(unsigned lds_addr) {
    int v;
    asm volatile("ds_read_b32 %0, %1\n\ts_waitcnt lgkmcnt(0)" : "=&v"(v) : "v"(lds_addr) : "memory");
    return v;
}
__device__ __forceinline__ int lds_min4_asm(unsigned lds_addr) {
    u32x4 v;
    asm volatile("ds_read_b128 %0, %1\n\ts_waitcnt lgkmcnt(0)" : "=&v"(v) : "v"(lds_addr) : "memory");
    const unsigned a = v.x < v.y ? v.x : v.y, b = v.z < v.w ? v.z : v.w;
    return __builtin_amdgcn_readfirstlane((int)(a < b ? a : b));
}
__device__ __forceinline__ void lds_st_asm(unsigned lds_addr, int v) {
    asm volatile("ds_write_b32 %0, %1" ::"v"(lds_addr), "v"(v) : "memory");
}
__device__ __forceinline__ int vm_outstanding() {
    unsigned r;
    asm volatile("s_getreg_b32 %0, hwreg(HW_REG_IB_STS)" : "=s"(r));
    return (int)((r & 15u) | ((r >> 18) & 48u));
}
__device__ __forceinline__ void lds_st(volatile int* p, int v) {
    __hip_atomic_store((int*)p, v, __ATOMIC_RELAXED, __HIP_MEMORY_SCOPE_WORKGROUP);
}
__device__ __forceinline__ void st_sys(void* p, unsigned v) {
    __hip_atomic_store((unsigned*)p, v, __ATOMIC_RELAXED, __HIP_MEMORY_SCOPE_SYSTEM);
}
__device__ __forceinline__ bool give_up(unsigned long long t0, unsigned* tmo, unsigned call) {
    const unsigned long long dt = __builtin_amdgcn_s_memrealtime() - t0;
    if (dt < SLOW_TICKS) return false;
    return dt > DEADLINE_TICKS || __builtin_amdgcn_readfirstlane(ld_sys_asm(tmo)) == call;
}
__device__ __forceinline__ __amdgpu_buffer_rsrc_t mk_rsrc(void* base, unsigned bytes) {
    return __builtin_amdgcn_make_buffer_rsrc(base, 0, bytes, 0x00020000);
}
__device__ __forceinline__ void st128(u32x4 v, __amdgpu_buffer_rsrc_t r, unsigned voff, int soff, int) {
    __builtin_amdgcn_raw_buffer_store_b128(v, r, voff, soff, 17);
    asm volatile("s_nop 1" ::"v"(v) : "memory");
}
__device__ __forceinline__ unsigned umax(unsigned a, unsigned b) { return a > b ? a : b; }
__device__ __forceinline__ unsigned wave_umax(unsigned v) {
    v = umax(v, (unsigned)__builtin_amdgcn_update_dpp(0, (int)v, 0xB1, 0xF, 0xF, false));
    v = umax(v, (unsigned)__builtin_amdgcn_update_dpp(0, (int)v, 0x4E, 0xF, 0xF, false));
    v = umax(v, (unsigned)__builtin_amdgcn_update_dpp(0, (int)v, 0x141, 0xF, 0xF, false));
    v = umax(v, (unsigned)__builtin_amdgcn_update_dpp(0, (int)v, 0x140, 0xF, 0xF, false));
    const unsigned a = __builtin_amdgcn_readlane(v, 0), b = __builtin_amdgcn_readlane(v, 16);
    const unsigned c = __builtin_amdgcn_readlane(v, 32), d = __builtin_amdgcn_readlane(v, 48);
    return umax(umax(a, b), umax(c, d));
}
__device__ __forceinline__ unsigned exp_ref(const float* sp) {
    const f32x4 v = *(const f32x4*)sp;
    const unsigned m = umax(umax(__float_as_uint(v.x) & 0x7FFFFFFFu, __float_as_uint(v.y) & 0x7FFFFFFFu),
                            umax(__float_as_uint(v.z) & 0x7FFFFFFFu, __float_as_uint(v.w) & 0x7FFFFFFFu));
    const unsigned r = ((wave_umax(m) >> 23) + 1u) >> 1;
    return r > 127u ? 127u : r;
}
__device__ __forceinline__ void enc8(const unsigned (&u)[4], unsigned R4, unsigned& l0, unsigned& l1, unsigned& n, unsigned& h) {
    const unsigned ha = __builtin_amdgcn_perm(u[1], u[0], 0x07050301u), hb = __builtin_amdgcn_perm(u[3], u[2], 0x07050301u);
    l0 = __builtin_amdgcn_perm(u[1], u[0], 0x06040200u);
    l1 = __builtin_amdgcn_perm(u[3], u[2], 0x06040200u);
    const unsigned da = (R4 - (ha & 0x7F7F7F7Fu)) & 0x7F7F7F7Fu, db = (R4 - (hb & 0x7F7F7F7Fu)) & 0x7F7F7F7Fu;
    n = (da & 0x07070707u) | ((ha >> 4) & 0x08080808u) | ((db & 0x07070707u) << 4) | (hb & 0x80808080u);
    h = ((da >> 3) & 0x0F0F0F0Fu) | ((db << 1) & 0xF0F0F0F0u);
}
__device__ __forceinline__ u32x4 dec8(unsigned l0, unsigned l1, unsigned n, unsigned h, unsigned R4) {
    const unsigned da = (n & 0x07070707u) | ((h & 0x0F0F0F0Fu) << 3);
    const unsigned db = ((n >> 4) & 0x07070707u) | ((h >> 1) & 0x78787878u);
    const unsigned ha = ((R4 - da) & 0x7F7F7F7Fu) | ((n << 4) & 0x80808080u);
    const unsigned hb = ((R4 - db) & 0x7F7F7F7Fu) | (n & 0x80808080u);
    return u32x4{__builtin_amdgcn_perm(ha, l0, 0x05010400u), __builtin_amdgcn_perm(ha, l0, 0x07030602u),
                 __builtin_amdgcn_perm(hb, l1, 0x05010400u), __builtin_amdgcn_perm(hb, l1, 0x07030602u)};
}
__device__ __forceinline__ void put_block(const unsigned (&ua)[4], const unsigned (&ub)[4], unsigned R4, int lane,
                                          __amdgpu_buffer_rsrc_t rd, __amdgpu_buffer_rsrc_t rh, int blk) {
    u32x4 l; unsigned n0, n1, h0, h1, la, lb;
    enc8(ua, R4, la, lb, n0, h0); l.x = la; l.y = lb;
    enc8(ub, R4, la, lb, n1, h1); l.z = la; l.w = lb;
    const unsigned long long hz = __ballot((h0 | h1) != 0u);
    if (hz) {
        const unsigned g0 = (unsigned)__builtin_amdgcn_update_dpp(0, (int)h0, 0xB1, 0xF, 0xF, false);
        const unsigned g1 = (unsigned)__builtin_amdgcn_update_dpp(0, (int)h1, 0xB1, 0xF, 0xF, false);
        if (!(lane & 1) && ((hz >> (lane & AGK_HG)) & AGK_HM))
            st128(u32x4{h0, h1, g0, g1}, rh, lane * 8, blk * 512, 17);
    }
    st128(l, rd, lane * 16, blk * 1536, 17);
    const unsigned m0 = (unsigned)__builtin_amdgcn_update_dpp(0, (int)n0, 0xB1, 0xF, 0xF, false);
    const unsigned m1 = (unsigned)__builtin_amdgcn_update_dpp(0, (int)n1, 0xB1, 0xF, 0xF, false);
    if (!(lane & 1)) st128(u32x4{n0, n1, m0, m1}, rd, 1024 + lane * 8, blk * 1536, 17);
}
#define STAMPW(i, w) do {} while (0)
#define STAMP(i) STAMPW(i, 0)

__global__ void __launch_bounds__(NTHR) ag_gemm_kernel(Params p) {
    __shared__ __attribute__((aligned(16))) char smem[4 * 32768 + 512];
    const int rank = p.rank;
    char* me = p.ws[rank];
    const int tid = threadIdx.x, lane = tid & 63;
    const int wv = __builtin_amdgcn_readfirstlane(tid >> 6);
    const int bid = blockIdx.x;
    const int Q = rank >> 1;
    const unsigned xswz = (unsigned)((lane >> 3) * 128 + (((lane & 7) ^ (lane >> 3)) * 16));
    STAMP(0);
    const unsigned callv = __hip_atomic_load((const unsigned*)(me + CALLW), __ATOMIC_RELAXED, __HIP_MEMORY_SCOPE_AGENT);
    unsigned* tmo = (unsigned*)(me + TMOW);

    if (bid > 0 && bid < PB0) {
        const unsigned call = __builtin_amdgcn_readfirstlane(callv) + 1u;
        const __amdgpu_buffer_rsrc_t rz = mk_rsrc(me + H_OFF + (size_t)((call + 1u) & 1u) * H_BYTES, (unsigned)H_BYTES);
        for (unsigned o = (unsigned)((bid - 1) * NTHR + tid) * 16u; o < (unsigned)H_BYTES; o += (PB0 - 1) * NTHR * 16)
            st128(u32x4{0u, 0u, 0u, 0u}, rz, o, 0, 17);
        wait_vm<0>();
        bar();
        if (tid == 0)
            __hip_atomic_store((unsigned long long*)(me + ZFLAG + bid * 128), (unsigned long long)call << 32, __ATOMIC_RELAXED, __HIP_MEMORY_SCOPE_AGENT);
        return;
    }
    if (bid >= RB0) {
        typedef unsigned u32x2 __attribute__((ext_vector_type(2)));
        const unsigned call = __builtin_amdgcn_readfirstlane(callv) + 1u;
        const int e = (bid - RB0) % 7;
        const int src = e < 6 ? 2 * ((Q + 1 + (e >> 1)) & 3) + (e & 1) : rank ^ 1;
        const __amdgpu_buffer_rsrc_t rd = mk_rsrc(me + D_OFF, (unsigned)D_BYTES);
        const __amdgpu_buffer_rsrc_t rh = mk_rsrc(me + H_OFF + (size_t)(call & 1u) * H_BYTES, (unsigned)H_BYTES);
        const __amdgpu_buffer_rsrc_t ro = mk_rsrc(me + A_OFF, (unsigned)(A_BYTES + B_BYTES));
        const char* pf = me + PFLAG + (size_t)src * (NITEMS * 128);
        char* rf = me + IFLAG + (size_t)src * (NITEMS * 128);
        for (int t = NPL0 * NSL + ((bid - RB0) / 7) * 9 + wv; t < NPL * NSL; t += RWG * 9) {
            const unsigned long long t0 = __builtin_amdgcn_s_memrealtime();
            unsigned fv;
            while (true) {
                fv = __builtin_amdgcn_readfirstlane(ld_sys_asm(pf + t * 128));
                if ((fv >> 14) == (call & 0x3FFFFu)) break;
                if (give_up(t0, tmo, call)) {
                    if (lane == 0) __hip_atomic_store(tmo, call, __ATOMIC_RELAXED, __HIP_MEMORY_SCOPE_AGENT);
                    return;
                }
                __builtin_amdgcn_s_sleep(2);
            }
            const unsigned Rx = (((fv >> 7) & 127u) | 0x80u) * 0x01010101u, Rw = ((fv & 127u) | 0x80u) * 0x01010101u;
            const int st = t >> 2, sl = t & 3;
            u32x4 l[5];
            u32x2 n[5], h[5];
            int blk[5];
#pragma unroll
            for (int i = 0; i < 5; ++i) {
                blk[i] = e == 6 ? 20 * (src & 1) + 5 * sl + i : i == 0 ? 40 + 4 * e + sl : 64 + ((2 + e) * 4 + i - 1) * 4 + sl;
                const int b = st * BPS + blk[i];
                l[i] = __builtin_amdgcn_raw_buffer_load_b128(rd, lane * 16, b * 1536, 17);
                n[i] = __builtin_amdgcn_raw_buffer_load_b64(rd, 1024 + lane * 8, b * 1536, 17);
                h[i] = __builtin_amdgcn_raw_buffer_load_b64(rh, lane * 8, b * 512, 17);
            }
#pragma unroll
            for (int i = 0; i < 5; ++i) {
                const bool isx = blk[i] < 64;
                const unsigned R = isx ? Rx : Rw;
                const int wb = blk[i] - 64;
                const int so = isx ? (st * AROWS + blk[i] * 16) * (BK * 2)
                                   : (int)A_BYTES + (st * (8 * BCOLS) + (wb >> 2) * 512 + (wb & 3) * 64) * 16;
                const unsigned vo = isx ? xswz : (unsigned)lane * 16u;
                st128(dec8(l[i].x, l[i].y, n[i].x, h[i].x, R), ro, vo, so, 17);
                st128(dec8(l[i].z, l[i].w, n[i].y, h[i].y, R), ro, vo, so + (isx ? 1024 : 4096), 17);
            }
            wait_vm<0>();
            if (lane == 0) st_sys(rf + t * 128, call);
        }
        return;
    }

    if (bid == 0) {
        if (wv >= 3) return;
        const unsigned call = __builtin_amdgcn_readfirstlane(callv) + 1u;
        volatile int* ctl = (volatile int*)(smem + 4 * 32768);
        char* gws = p.ws[0];
#pragma unroll
        for (int i = 1; i < NR; ++i) gws = (lane & 7) == i ? p.ws[i] : gws;
        unsigned long long* gdst = (unsigned long long*)(gws + GGRAN + ((call & 1u) * NR + rank) * 128);
        bool bad = false;
        {
            const unsigned long long* lg = (const unsigned long long*)(wv < 2 ? me + LGRAN + tid * 128 : me + ZFLAG + (1 + lane % (PB0 - 1)) * 128);
            const unsigned long long t0 = __builtin_amdgcn_s_memrealtime();
            unsigned long long v;
            while (true) {
                v = __hip_atomic_load(lg, __ATOMIC_RELAXED, __HIP_MEMORY_SCOPE_AGENT);
                if (__all((unsigned)(v >> 32) == call)) break;
                if (give_up(t0, tmo, call)) { bad = true; break; }
                __builtin_amdgcn_s_sleep(1);
            }
            unsigned am = wave_umax((unsigned)v);
            if (lane == 0) lds_st(ctl + wv, (int)am);
            __syncthreads();
            am = umax((unsigned)lds_ld(ctl), (unsigned)lds_ld(ctl + 1));
            if (tid < NR)
                __hip_atomic_store(gdst, ((unsigned long long)call << 32) | am, __ATOMIC_RELAXED, __HIP_MEMORY_SCOPE_SYSTEM);
        }
        STAMP(1);
        {
            const bool need = tid < 128 && (tid >> 5) != Q;
            const unsigned* df = (const unsigned*)(me + DFLAG + (tid & 127) * 128);
            const unsigned long long t0 = __builtin_amdgcn_s_memrealtime();
            while (true) {
                const unsigned v = __hip_atomic_load(df, __ATOMIC_RELAXED, __HIP_MEMORY_SCOPE_SYSTEM);
                if (__all(!need || v >= call)) break;
                if (give_up(t0, tmo, call)) { bad = true; break; }
                __builtin_amdgcn_s_sleep(1);
            }
            if (bad && lane == 0) {
                __hip_atomic_store(tmo, call, __ATOMIC_RELAXED, __HIP_MEMORY_SCOPE_AGENT);
                p.out[rank][(size_t)(2 * Q) * MS * NS] = 0x7F;
            }
            __syncthreads();
            if (tid == 0)
                __hip_atomic_store((unsigned*)(me + CALLW), call, __ATOMIC_RELAXED, __HIP_MEMORY_SCOPE_AGENT);
            if (tid < NSTREAM)
                __hip_atomic_store((unsigned*)(me + TICKW + tid * 128), 0u, __ATOMIC_RELAXED, __HIP_MEMORY_SCOPE_AGENT);
        }
        STAMP(2);
        return;
    }

    if (bid < CB0) {
        const int stream = (bid - PB0) >> 3, j0 = bid & 7;
        if (wv >= 7 || (stream >= 8 && wv == 0)) return;
        volatile int* tk = (volatile int*)smem;
        const unsigned tk_a = (unsigned)(uintptr_t)(__attribute__((address_space(3))) char*)smem;
        int dst = rank, fidx = rank;
        bool isx = true;
        int xrow = 16 * wv, arow = 320 * (rank & 1) + 16 * wv, xrs = 80;
        int wcol = 0, slot = 0;
        if (stream < 6) {
            const int i = stream >> 1, m = stream & 1;
            const int sl = 2 * ((2 - i) & 3) + (rank & 1);
            dst = 2 * ((Q + 1 + i) & 3) + m;
            isx = wv == 0;
            xrow = 320 + 64 * i; arow = 640 + 64 * sl; xrs = 16;
            wcol = 256 * m; slot = 2 + sl;
        } else if (stream == 6) {
            dst = rank ^ 1;
        } else if (stream >= 8) {
            isx = false; fidx = stream; wcol = 256 * (stream - 8); slot = stream - 8;
        }
        char* dws = p.ws[dst & 7];
        const float* xsmp = p.x + (size_t)(8 * lane) * KDIM + ((lane * 61) & 1023) * 4;
        const float* wsmp = p.w + (size_t)(64 * lane) * NS + ((lane * 7) & 127) * 4;
        const unsigned npl = stream < 7 ? (NPL - NPL0) * NSL : 0;
        auto planes = [&](int t) { return (unsigned)(t - NPL0 * NSL) < npl; };
        if (wv == 6) {
            unsigned* tc = (unsigned*)(me + TICKW + stream * 128);
            int t1 = j0 + 8;
            for (int i = 0;; ++i) {
                int v = NITEMS;
                if (t1 < NITEMS) {
                    if (lane == 0) v = (int)__hip_atomic_fetch_add(tc, 1u, __ATOMIC_RELAXED, __HIP_MEMORY_SCOPE_AGENT) + 16;
                    v = __builtin_amdgcn_readfirstlane(v);
                    if (lane == 0) lds_st(tk + ((i + 2) & 7), v);
                    asm volatile("s_waitcnt lgkmcnt(0)" ::: "memory");
                }
                bar();
                if (t1 >= NITEMS) break;
                t1 = v;
            }
            bar();
            return;
        }
        if (wv == 5) {
            const unsigned call = __builtin_amdgcn_readfirstlane(callv) + 1u;
            const unsigned pcall = (call << 14) | (exp_ref(xsmp) << 7) | exp_ref(wsmp);
            char* fb = dws + IFLAG + (size_t)fidx * (NITEMS * 128);
            char* pb = dws + PFLAG + (size_t)rank * (NITEMS * 128);
            auto publish = [&](int t) { if (lane == 0) st_sys((planes(t) ? pb : fb) + t * 128, planes(t) ? pcall : call); };
            int tp = -1, t0 = j0, t1 = j0 + 8;
            for (int i = 0;; ++i) {
                bar();
                if (tp >= 0) publish(tp);
                if (t1 >= NITEMS) break;
                tp = t0; t0 = t1;
                t1 = __builtin_amdgcn_readfirstlane(lds_ld_asm(tk_a + ((i + 2) & 7) * 4));
            }
            bar();
            publish(t0);
            return;
        }
        const __amdgpu_buffer_rsrc_t rd = mk_rsrc(dws + D_OFF, (unsigned)D_BYTES);
        __amdgpu_buffer_rsrc_t rh;
        unsigned R4;
        auto setup = [&](const float* smp) {
            R4 = (exp_ref(smp) | 0x80u) * 0x01010101u;
            rh = mk_rsrc(dws + H_OFF + (size_t)((__builtin_amdgcn_readfirstlane(callv) + 1u) & 1u) * H_BYTES, (unsigned)H_BYTES);
        };
        if (isx) {
            const __amdgpu_buffer_rsrc_t r0 = mk_rsrc(dws + A_OFF, (unsigned)A_BYTES);
            const unsigned vo0 = (unsigned)((xrow + (lane >> 3)) * KDIM + (lane & 7) * 8) * 4u, vo1 = vo0 + 8 * KDIM * 4;
            const unsigned doff = (unsigned)(arow * BK * 2) + xswz;
            f32x4 va[4], vb[4];
            auto ld = [&](f32x4 (&v)[4], int t) {
                const float* sp = p.x + (size_t)((t & 3) * xrs) * KDIM + (t >> 2) * BK;
                asm volatile("s_nop 4\n\t"
                             "global_load_dwordx4 %0, %4, %6\n\tglobal_load_dwordx4 %1, %4, %6 offset:16\n\t"
                             "global_load_dwordx4 %2, %5, %6\n\tglobal_load_dwordx4 %3, %5, %6 offset:16"
                             : "=&v"(v[0]), "=&v"(v[1]), "=&v"(v[2]), "=&v"(v[3]) : "v"(vo0), "v"(vo1), "s"(sp) : "memory");
            };
            auto step = [&](f32x4 (&cur)[4], f32x4 (&nxt)[4], int t, int tn) {
                ld(nxt, tn < NITEMS ? tn : t);
                asm volatile("s_waitcnt vmcnt(6)" : "+v"(cur[0]), "+v"(cur[1]), "+v"(cur[2]), "+v"(cur[3]) :: "memory");
                const unsigned ua[4] = {pk_bf16(cur[0].x, cur[0].y), pk_bf16(cur[0].z, cur[0].w), pk_bf16(cur[1].x, cur[1].y), pk_bf16(cur[1].z, cur[1].w)};
                const unsigned ub[4] = {pk_bf16(cur[2].x, cur[2].y), pk_bf16(cur[2].z, cur[2].w), pk_bf16(cur[3].x, cur[3].y), pk_bf16(cur[3].z, cur[3].w)};
                if (planes(t)) {
                    put_block(ua, ub, R4, lane, rd, rh, (t >> 2) * BPS + ((arow + (t & 3) * xrs) >> 4));
                } else {
                    const int so = ((t >> 2) * AROWS + (t & 3) * xrs) * (BK * 2);
                    st128(u32x4{ua[0], ua[1], ua[2], ua[3]}, r0, doff, so, 17);
                    st128(u32x4{ub[0], ub[1], ub[2], ub[3]}, r0, doff + 1024, so, 17);
                }
                wait_vm<6>();
                bar();
            };
            int t = j0, tn = j0 + 8;
            ld(va, t);
            setup(xsmp);
            asm volatile("s_waitcnt vmcnt(0)" : "+v"(va[0]), "+v"(va[1]), "+v"(va[2]), "+v"(va[3]) :: "memory");
            for (int i = 0;; i += 2) {
                step(va, vb, t, tn);
                if (tn >= NITEMS) break;
                t = tn; tn = __builtin_amdgcn_readfirstlane(lds_ld_asm(tk_a + ((i + 2) & 7) * 4));
                step(vb, va, t, tn);
                if (tn >= NITEMS) break;
                t = tn; tn = __builtin_amdgcn_readfirstlane(lds_ld_asm(tk_a + ((i + 3) & 7) * 4));
            }
            asm volatile("s_waitcnt vmcnt(0)" : "+v"(va[0]), "+v"(va[1]), "+v"(va[2]), "+v"(va[3]),
                                                "+v"(vb[0]), "+v"(vb[1]), "+v"(vb[2]), "+v"(vb[3]) :: "memory");
        } else {
            const __amdgpu_buffer_rsrc_t r0 = mk_rsrc(dws + B_OFF, (unsigned)B_BYTES);
            const int col = ((lane >> 2) & 3) * 16 + (lane >> 4) * 4 + (lane & 3);
            const unsigned vo0 = (unsigned)(((wv - 1) * 16 + 2) * NS + wcol + col) * 4u;
            const unsigned vo1 = vo0 + 4 * NS * 4, vo2 = vo0 + 8 * NS * 4, vo3 = vo0 + 12 * NS * 4;
            const unsigned doff = (unsigned)((slot * 8 + (wv - 1) * 2) * 256 + lane) * 16u;
            float va[16], vb[16];
#define AGK_V16(c, v) c(v[0]), c(v[1]), c(v[2]), c(v[3]), c(v[4]), c(v[5]), c(v[6]), c(v[7]), \
                      c(v[8]), c(v[9]), c(v[10]), c(v[11]), c(v[12]), c(v[13]), c(v[14]), c(v[15])
            auto ld = [&](float (&v)[16], int t) {
                const float* sp = p.w + (size_t)((t >> 2) * BK) * NS + (t & 3) * 64;
                asm volatile("s_nop 4\n\t"
                             "global_load_dword %0, %16, %20 offset:-4096\n\tglobal_load_dword %1, %16, %20 offset:-2048\n\t"
                             "global_load_dword %2, %16, %20\n\tglobal_load_dword %3, %16, %20 offset:2048\n\t"
                             "global_load_dword %4, %17, %20 offset:-4096\n\tglobal_load_dword %5, %17, %20 offset:-2048\n\t"
                             "global_load_dword %6, %17, %20\n\tglobal_load_dword %7, %17, %20 offset:2048\n\t"
                             "global_load_dword %8, %18, %20 offset:-4096\n\tglobal_load_dword %9, %18, %20 offset:-2048\n\t"
                             "global_load_dword %10, %18, %20\n\tglobal_load_dword %11, %18, %20 offset:2048\n\t"
                             "global_load_dword %12, %19, %20 offset:-4096\n\tglobal_load_dword %13, %19, %20 offset:-2048\n\t"
                             "global_load_dword %14, %19, %20\n\tglobal_load_dword %15, %19, %20 offset:2048"
                             : AGK_V16("=&v", v) : "v"(vo0), "v"(vo1), "v"(vo2), "v"(vo3), "s"(sp) : "memory");
            };
            auto step = [&](float (&cur)[16], float (&nxt)[16], int t, int tn) {
                ld(nxt, tn < NITEMS ? tn : t);
                asm volatile("s_waitcnt vmcnt(18)" : AGK_V16("+v", cur) :: "memory");
                const unsigned ua[4] = {pk_bf16(cur[0], cur[1]), pk_bf16(cur[2], cur[3]), pk_bf16(cur[4], cur[5]), pk_bf16(cur[6], cur[7])};
                const unsigned ub[4] = {pk_bf16(cur[8], cur[9]), pk_bf16(cur[10], cur[11]), pk_bf16(cur[12], cur[13]), pk_bf16(cur[14], cur[15])};
                if (planes(t)) {
                    put_block(ua, ub, R4, lane, rd, rh, (t >> 2) * BPS + 64 + (slot * 4 + (wv - 1)) * 4 + (t & 3));
                } else {
                    const int so = ((t >> 2) * (8 * BCOLS) + (t & 3) * 64) * 16;
                    st128(u32x4{ua[0], ua[1], ua[2], ua[3]}, r0, doff, so, 17);
                    st128(u32x4{ub[0], ub[1], ub[2], ub[3]}, r0, doff + 256 * 16, so, 17);
                }
                wait_vm<18>();
                bar();
            };
            int t = j0, tn = j0 + 8;
            ld(va, t);
            setup(wsmp);
            asm volatile("s_waitcnt vmcnt(0)" : AGK_V16("+v", va) :: "memory");
            for (int i = 0;; i += 2) {
                step(va, vb, t, tn);
                if (tn >= NITEMS) break;
                t = tn; tn = __builtin_amdgcn_readfirstlane(lds_ld_asm(tk_a + ((i + 2) & 7) * 4));
                step(vb, va, t, tn);
                if (tn >= NITEMS) break;
                t = tn; tn = __builtin_amdgcn_readfirstlane(lds_ld_asm(tk_a + ((i + 3) & 7) * 4));
            }
            asm volatile("s_waitcnt vmcnt(0)" : AGK_V16("+v", va) :: "memory");
            asm volatile("" : AGK_V16("+v", vb) :: "memory");
        }
        wait_vm<0>();
        bar();
        STAMPW(1, 1);
        return;
    }

    const int cid = bid - CB0;
    const int xg = cid & 7, xi = cid >> 3;
    const int rb = (xg & 1) * 4 + (xi & 3);
    const int cb = (xg >> 1) * 4 + (xi >> 2);
    volatile int* ctl = (volatile int*)(smem + 4 * 32768);
    const unsigned ctl_a = (unsigned)(uintptr_t)(__attribute__((address_space(3))) char*)(smem + 4 * 32768);
    if (tid < 16) lds_st(ctl + tid, 0);
    __syncthreads();
    const unsigned call = __builtin_amdgcn_readfirstlane(callv) + 1u;
    auto grp_src = [&](int g) { return g < 5 ? 2 * Q : g < 10 ? 2 * Q + 1 : 2 * ((Q + 1 + ((g - 10) >> 1)) & 3) + ((g - 10) & 1); };
    const int csrc = (cb >> 1) < 2 ? 8 + (cb >> 1) : 2 * ((Q + 1 + (((cb >> 1) - 2) >> 1)) & 3) + (((cb >> 1) - 2) & 1);

    if (wv == 8) {
        const int psel = (lane >> 2) & 3;
        const int psrc = psel == 0 ? grp_src(2 * rb) : psel == 1 ? grp_src(2 * rb + 1) : csrc;
        const char* pfb = me + IFLAG + (size_t)psrc * (NITEMS * 128) + (lane & 3) * 128;
        int ready = 0, base = 0;
        unsigned dm = 0;
        bool bad = false;
        unsigned long long t0 = __builtin_amdgcn_s_memrealtime();
        while (ready < NST) {
            unsigned v[PWN];
            const char* pa[PWN];
#pragma unroll
            for (int i = 0; i < PWN; ++i) {
                const int c = base + 4 * i + (lane >> 4);
                pa[i] = pfb + (c < NST ? c : NST - 1) * (NSL * 128);
            }
#pragma unroll
            for (int i = 0; i < PWN; ++i)
                asm volatile("global_load_dword %0, %1, off sc0 sc1" : "=&v"(v[i]) : "v"(pa[i]) : "memory");
#pragma unroll
            for (int i = 0; i < PWN; ++i) asm volatile("s_waitcnt vmcnt(0)" : "+v"(v[i]) :: "memory");
            const int r0 = ready;
#pragma unroll
            for (int i = 0; i < PWN; ++i) {
                const unsigned long long ok = __ballot(v[i] == call);
#pragma unroll
                for (int o = 0; o < 4; ++o) {
                    const int d = 4 * i + o;
                    if (((unsigned)(ok >> (16 * o)) & 0xFFFFu) == 0xFFFFu && base + d < NST && !((dm >> d) & 1u)) {
                        lds_st_asm(ctl_a + 64 + ready * 4, base + d);
                        ++ready;
                        dm |= 1u << d;
                    }
                }
            }
            if (ready != r0) {
                lds_st_asm(ctl_a, ready);
                const int n = __builtin_ctz(~dm);
                base += n; dm >>= n;
                continue;
            }
            if (give_up(t0, tmo, call)) {
                bad = true;
                for (; base < NST; ++base, dm >>= 1)
                    if (!(dm & 1u)) { if (lane == 0) lds_st(ctl + 16 + ready, base); ++ready; }
                if (lane == 0) lds_st(ctl, ready);
                break;
            }
            __builtin_amdgcn_s_sleep(1);
        }
        const unsigned long long* gp = (const unsigned long long*)(me + GGRAN + ((call & 1u) * NR + (lane & 7)) * 128);
        t0 = __builtin_amdgcn_s_memrealtime();
        unsigned long long v;
        while (true) {
            v = __hip_atomic_load(gp, __ATOMIC_RELAXED, __HIP_MEMORY_SCOPE_SYSTEM);
            if (__all((unsigned)(v >> 32) == call)) break;
            if (give_up(t0, tmo, call)) { bad = true; break; }
            __builtin_amdgcn_s_sleep(1);
        }
        const unsigned am = wave_umax((unsigned)v);
        if (lane == 0) {
            if (bad) {
                __hip_atomic_store(tmo, call, __ATOMIC_RELAXED, __HIP_MEMORY_SCOPE_AGENT);
                lds_st(ctl + 1, 1);
            }
            lds_st(ctl + 12, (int)am);
        }
        asm volatile("s_waitcnt lgkmcnt(0)" ::: "memory");
        return;
    }

    if (wv >= 4) {
        const int lw = wv - 4;
        const char* xsrc = me + A_OFF + (size_t)(rb * 128 + lw * 32) * (BK * 2) + lane * 16;
        const char* wsrc = me + B_OFF + (size_t)(((cb >> 1) * 8 + lw * 2) * 256 + (cb & 1) * 128 + lane) * 16;
        int issued = 0, landed = 0;
        auto retire = [&]() {
            const int out = issued - landed;
            if (out == 1) wait_vm<0>(); else if (out == 2) wait_vm<8>(); else if (out == 3) wait_vm<16>(); else wait_vm<24>();
            ++landed;
            lds_st_asm(ctl_a + 16 + lw * 4, landed);
        };
        while (issued < NST) {
            unsigned cnt; u32x4 fw; unsigned long long qe;
            asm volatile("ds_read_b32 %0, %3\n\tds_read_b128 %1, %3 offset:32\n\tds_read2_b32 %2, %4 offset0:0 offset1:1\n\ts_waitcnt lgkmcnt(0)"
                         : "=&v"(cnt), "=&v"(fw), "=&v"(qe) : "v"(ctl_a), "v"(ctl_a + 64 + issued * 4) : "memory");
            const unsigned fa = fw.x < fw.y ? fw.x : fw.y, fb = fw.z < fw.w ? fw.z : fw.w;
            const int fr = __builtin_amdgcn_readfirstlane((int)(fa < fb ? fa : fb)) + 4;
            const int rdy = __builtin_amdgcn_readfirstlane((int)cnt);
            const int lim = rdy < fr ? rdy : fr;
            bool progress = false;
#pragma unroll
            for (int k = 0; k < 2; ++k) {
                if (issued < lim) {
                    const int st = __builtin_amdgcn_readfirstlane((int)(k == 0 ? (unsigned)qe : (unsigned)(qe >> 32))) & (NST - 1);
                    char* xb = smem + (issued & 3) * 32768 + lw * 4096;
                    char* wb = xb + 16384;
                    const char* xs = xsrc + (size_t)st * (AROWS * BK * 2);
                    const char* wsn = wsrc + (size_t)st * (8 * BCOLS * 16);
#pragma unroll
                    for (int i = 0; i < 4; ++i)
                        __builtin_amdgcn_global_load_lds((const unsigned*)(xs + i * 1024), (unsigned*)(xb + i * 1024), 16, 0, 17);
#pragma unroll
                    for (int i = 0; i < 4; ++i)
                        __builtin_amdgcn_global_load_lds((const unsigned*)(wsn + (i >> 1) * (256 * 16) + (i & 1) * 1024), (unsigned*)(wb + i * 1024), 16, 0, 17);
                    ++issued;
                    progress = true;
                }
                while (issued > landed && (issued - landed == 4 || vm_outstanding() <= 8 * (issued - landed - 1))) { retire(); progress = true; }
            }
            if (!progress) __builtin_amdgcn_s_sleep(1);
        }
        while (landed < NST) retire();
        return;
    }

    const int wm = wv >> 1, wn = wv & 1;
    const int ml = lane & 15, q = lane >> 4;
    f32x4 acc[4][4];
#pragma unroll
    for (int u = 0; u < 4; ++u)
#pragma unroll
        for (int t = 0; t < 4; ++t) acc[u][t] = f32x4{0.f, 0.f, 0.f, 0.f};
    const int xrd = (wm * 64 + ml) * 128;
    const int wrd = 16384 + (q * 128 + wn * 64 + ml) * 16;
    STAMP(1);
    bf16x8 wf0[4], xf0[4], wf1[4], xf1[4];
#pragma unroll
    for (int t = 0; t < 4; ++t) { wf1[t] = bf16x8{}; xf1[t] = bf16x8{}; }
    const int xsw0 = (q ^ (ml & 7)) * 16, xsw1 = ((4 + q) ^ (ml & 7)) * 16;
    int full = 0;
    for (int s = 0; s < NST; ++s) {
        while (full <= s) {
            full = lds_min4_asm(ctl_a + 16);
            if (full <= s) __builtin_amdgcn_s_sleep(1);
        }
        const char* sb = smem + (s & 3) * 32768;
#pragma unroll
        for (int t = 0; t < 4; ++t) wf0[t] = *(const bf16x8*)(sb + wrd + t * 256);
#pragma unroll
        for (int u = 0; u < 4; ++u) xf0[u] = *(const bf16x8*)(sb + xrd + u * 2048 + xsw0);
        __builtin_amdgcn_sched_barrier(0);
#pragma unroll
        for (int u = 0; u < 4; ++u)
#pragma unroll
            for (int t = 0; t < 4; ++t)
                acc[u][t] = __builtin_amdgcn_mfma_f32_16x16x32_bf16(wf1[t], xf1[u], acc[u][t], 0, 0, 0);
        __builtin_amdgcn_sched_barrier(0);
        u32x4 ln;
        asm volatile("ds_read_b128 %0, %1" : "=&v"(ln) : "v"(ctl_a + 16) : "memory");
#pragma unroll
        for (int t = 0; t < 4; ++t) wf1[t] = *(const bf16x8*)(sb + wrd + (4 * 128 * 16) + t * 256);
#pragma unroll
        for (int u = 0; u < 4; ++u) xf1[u] = *(const bf16x8*)(sb + xrd + u * 2048 + xsw1);
        __builtin_amdgcn_sched_barrier(0);
#pragma unroll
        for (int u = 0; u < 2; ++u)
#pragma unroll
            for (int t = 0; t < 4; ++t)
                acc[u][t] = __builtin_amdgcn_mfma_f32_16x16x32_bf16(wf0[t], xf0[u], acc[u][t], 0, 0, 0);
        __builtin_amdgcn_sched_barrier(0);
        asm volatile("s_waitcnt lgkmcnt(0)" : "+v"(ln) :: "memory");
        lds_st_asm(ctl_a + 32 + wv * 4, s + 1);
        {
            const unsigned a = ln.x < ln.y ? ln.x : ln.y, b = ln.z < ln.w ? ln.z : ln.w;
            full = __builtin_amdgcn_readfirstlane((int)(a < b ? a : b));
        }
        __builtin_amdgcn_sched_barrier(0);
#pragma unroll
        for (int u = 2; u < 4; ++u)
#pragma unroll
            for (int t = 0; t < 4; ++t)
                acc[u][t] = __builtin_amdgcn_mfma_f32_16x16x32_bf16(wf0[t], xf0[u], acc[u][t], 0, 0, 0);
        __builtin_amdgcn_sched_barrier(0);
    }
#pragma unroll
    for (int u = 0; u < 4; ++u)
#pragma unroll
        for (int t = 0; t < 4; ++t)
            acc[u][t] = __builtin_amdgcn_mfma_f32_16x16x32_bf16(wf1[t], xf1[u], acc[u][t], 0, 0, 0);
    STAMP(3);

    float mx = 0.f;
#pragma unroll
    for (int u = 0; u < 4; ++u)
#pragma unroll
        for (int t = 0; t < 4; ++t)
            mx = fmaxf(fmaxf(mx, fmaxf(acc[u][t][0], acc[u][t][1])), fmaxf(acc[u][t][2], acc[u][t][3]));
    {
        const unsigned wmx = wave_umax(__float_as_uint(mx));
        int tk = 0;
        if (lane == 0) {
            __hip_atomic_fetch_max((unsigned*)(ctl + 2), wmx, __ATOMIC_RELAXED, __HIP_MEMORY_SCOPE_WORKGROUP);
            tk = __hip_atomic_fetch_add((int*)(ctl + 3), 1, __ATOMIC_RELAXED, __HIP_MEMORY_SCOPE_WORKGROUP);
        }
        tk = __builtin_amdgcn_readfirstlane(tk);
        if (tk == 3) {
            const unsigned tm = (unsigned)lds_ld(ctl + 2);
            if (lane == 0)
                __hip_atomic_store((unsigned long long*)(me + LGRAN + cid * 128), ((unsigned long long)call << 32) | tm,
                                   __ATOMIC_RELAXED, __HIP_MEMORY_SCOPE_AGENT);
        }
    }
    const int g = rb * 2 + wm;
    const int srk = grp_src(g);
    const int row0 = g < 5 ? 64 * g : g < 10 ? 64 * (g - 5) : 320 + 64 * ((2 - ((g - 10) >> 1)) & 3);
    const int owner = ((cb >> 1) < 2 ? rank : csrc) & 7;
    const int coff = (cb >> 1) < 2 ? cb * 128 : 256 * (rank & 1) + (cb & 1) * 128;
    const __amdgpu_buffer_rsrc_t ro = mk_rsrc(p.out[owner], (unsigned)(KDIM * NS));
    const unsigned ooff = (unsigned)((srk * MS + row0 + ml) * NS + coff + wn * 64 + q * 16);
    unsigned* dfl = (unsigned*)(p.ws[owner] + DFLAG + (rank * 16 + rb * 2 + (cb & 1)) * 128);
    bar();
    STAMP(4);
    mx = __int_as_float(lds_ld(ctl + 12));
    const bool bad = lds_ld(ctl + 1) != 0;
    const float inv = mx > 0.f ? 448.0f / mx : 0.f;
#pragma unroll
    for (int u = 0; u < 4; ++u) {
        u32x4 o;
#pragma unroll
        for (int t = 0; t < 4; ++t) {
            const float a0 = __builtin_amdgcn_fmed3f(acc[u][t][0] * inv, 0.f, 448.f), a1 = __builtin_amdgcn_fmed3f(acc[u][t][1] * inv, 0.f, 448.f);
            const float a2 = __builtin_amdgcn_fmed3f(acc[u][t][2] * inv, 0.f, 448.f), a3 = __builtin_amdgcn_fmed3f(acc[u][t][3] * inv, 0.f, 448.f);
            int r = __builtin_amdgcn_cvt_pk_fp8_f32(a0, a1, 0, false);
            r = __builtin_amdgcn_cvt_pk_fp8_f32(a2, a3, r, true);
            o[t] = bad ? 0x7F7F7F7Fu : (unsigned)r;
        }
        st128(o, ro, ooff, u * (16 * NS), 17);
    }
    wait_vm<0>();
    bar();
    if (tid == 0 && owner != rank) st_sys(dfl, call);
    STAMP(5);
}

}

extern "C" void kernel_launch(int rank, int nranks, ncclComm_t comm,
                              hipStream_t stream,
                              void* const* d_in, const int* in_sizes, int n_in,
                              void* d_out, int out_size,
                              const int* in_dtypes, int out_dtype,
                              void* const* d_out_all,
                              void* const* d_ws_all, size_t ws_bytes) {
    (void)comm; (void)in_sizes; (void)n_in; (void)d_out; (void)out_size; (void)in_dtypes; (void)out_dtype;
    if (nranks != agk::NR || (unsigned)rank >= (unsigned)agk::NR || ws_bytes < agk::WS_NEED) {
        fprintf(stderr, "unsupported configuration\n");
        return;
    }
    agk::Params p;
    p.x = (const float*)d_in[0];
    p.w = (const float*)d_in[1];
    for (int i = 0; i < agk::NR; ++i) {
        p.ws[i] = (char*)d_ws_all[i];
        p.out[i] = (unsigned char*)d_out_all[i];
    }
    p.rank = rank;
    hipLaunchKernelGGL(agk::ag_gemm_kernel, dim3(agk::GRID), dim3(agk::NTHR), 0, stream, p);
    HIP_CHECK(hipGetLastError());
}
```

```cpp
#include <hip/hip_runtime.h>
#include <rccl/rccl.h>
#include <cstdio>
#include <cstdint>

#define HIP_CHECK(expr) do { hipError_t _e = (expr); if (_e != hipSuccess) { \
    fprintf(stderr, "HIP error %s at %s:%d\n", hipGetErrorString(_e), __FILE__, __LINE__); \
    return; } } while (0)

namespace agk {

constexpr int NR = 8;
constexpr int KDIM = 4096, MS = 512, NS = 512;
constexpr int BK = 64;
constexpr int NST = KDIM / BK;
constexpr int AROWS = 1024, BCOLS = 2048;
constexpr int NSTREAM = 10;
constexpr int NSL = 4;
constexpr int NITEMS = NST * NSL;
constexpr int NCONS = 128;
constexpr int NPL0 = 0, NPL = NST - 4;
constexpr int NPOLL = 2;
constexpr int NWV = 8 + NPOLL, NTHR = NWV * 64;
constexpr int RWG = 5;
constexpr int PB0 = 8, CB0 = PB0 + NSTREAM * 8, RB0 = CB0 + NCONS, GRID = RB0 + 7 * RWG;

constexpr size_t A_OFF = 0;
constexpr size_t A_BYTES = (size_t)AROWS * KDIM * 2;
constexpr size_t B_OFF = A_OFF + A_BYTES;
constexpr size_t B_BYTES = (size_t)BCOLS * KDIM * 2;
constexpr int BPS = 192;
constexpr size_t D_OFF = B_OFF + B_BYTES;
constexpr size_t D_BYTES = (size_t)NST * BPS * 1536;
constexpr size_t H_OFF = D_OFF + D_BYTES;
constexpr size_t H_BYTES = (size_t)NST * BPS * 512;
constexpr size_t C_OFF = H_OFF + 2 * H_BYTES;
constexpr size_t DFLAG = C_OFF;
constexpr size_t LGRAN = C_OFF + 16384;
constexpr size_t GGRAN = C_OFF + 32768;
constexpr size_t TICKW = C_OFF + 36864;
constexpr size_t CALLW = C_OFF + 40960;
constexpr size_t TMOW = C_OFF + 40960 + 128;
constexpr size_t ZFLAG = C_OFF + 45056;
constexpr size_t TRACE = C_OFF + 65536;
constexpr size_t IFLAG = C_OFF + 262144;
constexpr size_t PFLAG = IFLAG + (size_t)NSTREAM * NITEMS * 128;
constexpr size_t WS_NEED = PFLAG + (size_t)NR * NITEMS * 128;
constexpr unsigned long long DEADLINE_TICKS = 50000000ull;
constexpr unsigned long long SLOW_TICKS = 20000ull;

typedef float f32x4 __attribute__((ext_vector_type(4)));
typedef float f32x2 __attribute__((ext_vector_type(2)));
typedef __bf16 bf16x2 __attribute__((ext_vector_type(2)));
typedef __bf16 bf16x8 __attribute__((ext_vector_type(8)));
typedef unsigned u32x4 __attribute__((ext_vector_type(4)));

struct Params {
    const float* x;
    const float* w;
    char* ws[NR];
    unsigned char* out[NR];
    int rank;
};

template <int N> __device__ __forceinline__ void wait_vm() {
    asm volatile("s_waitcnt vmcnt(%0)" ::"i"(N) : "memory");
}
__device__ __forceinline__ void bar() { asm volatile("s_barrier" ::: "memory"); }

__device__ __forceinline__ unsigned pk_bf16(float a, float b) {
    f32x2 v = {a, b};
    bf16x2 r = __builtin_convertvector(v, bf16x2);
    return __builtin_bit_cast(unsigned, r);
}
__device__ __forceinline__ unsigned ld_sys_asm(const void* p) {
    unsigned v;
    asm volatile("global_load_dword %0, %1, off sc0 sc1\n\ts_waitcnt vmcnt(0)" : "=&v"(v) : "v"(p) : "memory");
    return v;
}
__device__ __forceinline__ int lds_ld(volatile int* p) {
    return __hip_atomic_load((int*)p, __ATOMIC_RELAXED, __HIP_MEMORY_SCOPE_WORKGROUP);
}
__device__ __forceinline__ int lds_ld_asm(unsigned lds_addr) {
    int v;
    asm volatile("ds_read_b32 %0, %1\n\ts_waitcnt lgkmcnt(0)" : "=&v"(v) : "v"(lds_addr) : "memory");
    return v;
}
__device__ __forceinline__ int lds_min4_asm(unsigned lds_addr) {
    u32x4 v;
    asm volatile("ds_read_b128 %0, %1\n\ts_waitcnt lgkmcnt(0)" : "=&v"(v) : "v"(lds_addr) : "memory");
    const unsigned a = v.x < v.y ? v.x : v.y, b = v.z < v.w ? v.z : v.w;
    return __builtin_amdgcn_readfirstlane((int)(a < b ? a : b));
}
__device__ __forceinline__ void lds_st_asm(unsigned lds_addr, int v) {
    asm volatile("ds_write_b32 %0, %1" ::"v"(lds_addr), "v"(v) : "memory");
}
__device__ __forceinline__ int vm_outstanding() {
    unsigned r;
    asm volatile("s_getreg_b32 %0, hwreg(HW_REG_IB_STS)" : "=s"(r));
    return (int)((r & 15u) | ((r >> 18) & 48u));
}
__device__ __forceinline__ void lds_st(volatile int* p, int v) {
    __hip_atomic_store((int*)p, v, __ATOMIC_RELAXED, __HIP_MEMORY_SCOPE_WORKGROUP);
}
__device__ __forceinline__ void st_sys(void* p, unsigned v) {
    __hip_atomic_store((unsigned*)p, v, __ATOMIC_RELAXED, __HIP_MEMORY_SCOPE_SYSTEM);
}
__device__ __forceinline__ bool give_up(unsigned long long t0, unsigned* tmo, unsigned call) {
    const unsigned long long dt = __builtin_amdgcn_s_memrealtime() - t0;
    if (dt < SLOW_TICKS) return false;
    return dt > DEADLINE_TICKS || __builtin_amdgcn_readfirstlane(ld_sys_asm(tmo)) == call;
}
__device__ __forceinline__ __amdgpu_buffer_rsrc_t mk_rsrc(void* base, unsigned bytes) {
    return __builtin_amdgcn_make_buffer_rsrc(base, 0, bytes, 0x00020000);
}
__device__ __forceinline__ void st128(u32x4 v, __amdgpu_buffer_rsrc_t r, unsigned voff, int soff, int) {
    __builtin_amdgcn_raw_buffer_store_b128(v, r, voff, soff, 17);
    asm volatile("s_nop 1" ::"v"(v) : "memory");
}
__device__ __forceinline__ unsigned umax(unsigned a, unsigned b) { return a > b ? a : b; }
__device__ __forceinline__ unsigned wave_umax(unsigned v) {
    v = umax(v, (unsigned)__builtin_amdgcn_update_dpp(0, (int)v, 0xB1, 0xF, 0xF, false));
    v = umax(v, (unsigned)__builtin_amdgcn_update_dpp(0, (int)v, 0x4E, 0xF, 0xF, false));
    v = umax(v, (unsigned)__builtin_amdgcn_update_dpp(0, (int)v, 0x141, 0xF, 0xF, false));
    v = umax(v, (unsigned)__builtin_amdgcn_update_dpp(0, (int)v, 0x140, 0xF, 0xF, false));
    const unsigned a = __builtin_amdgcn_readlane(v, 0), b = __builtin_amdgcn_readlane(v, 16);
    const unsigned c = __builtin_amdgcn_readlane(v, 32), d = __builtin_amdgcn_readlane(v, 48);
    return umax(umax(a, b), umax(c, d));
}
__device__ __forceinline__ unsigned exp_ref(const float* sp) {
    const f32x4 v = *(const f32x4*)sp;
    const unsigned m = umax(umax(__float_as_uint(v.x) & 0x7FFFFFFFu, __float_as_uint(v.y) & 0x7FFFFFFFu),
                            umax(__float_as_uint(v.z) & 0x7FFFFFFFu, __float_as_uint(v.w) & 0x7FFFFFFFu));
    const unsigned r = ((wave_umax(m) >> 23) + 1u) >> 1;
    return r > 127u ? 127u : r;
}
__device__ __forceinline__ void enc8(const unsigned (&u)[4], unsigned R4, unsigned& l0, unsigned& l1, unsigned& n, unsigned& h) {
    const unsigned ha = __builtin_amdgcn_perm(u[1], u[0], 0x07050301u), hb = __builtin_amdgcn_perm(u[3], u[2], 0x07050301u);
    l0 = __builtin_amdgcn_perm(u[1], u[0], 0x06040200u);
    l1 = __builtin_amdgcn_perm(u[3], u[2], 0x06040200u);
    const unsigned da = (R4 - (ha & 0x7F7F7F7Fu)) & 0x7F7F7F7Fu, db = (R4 - (hb & 0x7F7F7F7Fu)) & 0x7F7F7F7Fu;
    n = (da & 0x07070707u) | ((ha >> 4) & 0x08080808u) | ((db & 0x07070707u) << 4) | (hb & 0x80808080u);
    h = ((da >> 3) & 0x0F0F0F0Fu) | ((db << 1) & 0xF0F0F0F0u);
}
__device__ __forceinline__ u32x4 dec8(unsigned l0, unsigned l1, unsigned n, unsigned h, unsigned R4) {
    const unsigned da = (n & 0x07070707u) | ((h & 0x0F0F0F0Fu) << 3);
    const unsigned db = ((n >> 4) & 0x07070707u) | ((h >> 1) & 0x78787878u);
    const unsigned ha = ((R4 - da) & 0x7F7F7F7Fu) | ((n << 4) & 0x80808080u);
    const unsigned hb = ((R4 - db) & 0x7F7F7F7Fu) | (n & 0x80808080u);
    return u32x4{__builtin_amdgcn_perm(ha, l0, 0x05010400u), __builtin_amdgcn_perm(ha, l0, 0x07030602u),
                 __builtin_amdgcn_perm(hb, l1, 0x05010400u), __builtin_amdgcn_perm(hb, l1, 0x07030602u)};
}
__device__ __forceinline__ void put_block(const unsigned (&ua)[4], const unsigned (&ub)[4], unsigned R4, int lane,
                                          __amdgpu_buffer_rsrc_t rd, __amdgpu_buffer_rsrc_t rh, int blk) {
    u32x4 l; unsigned n0, n1, h0, h1, la, lb;
    enc8(ua, R4, la, lb, n0, h0); l.x = la; l.y = lb;
    enc8(ub, R4, la, lb, n1, h1); l.z = la; l.w = lb;
    const unsigned long long hz = __ballot((h0 | h1) != 0u);
    if (hz) {
        const unsigned g0 = (unsigned)__builtin_amdgcn_update_dpp(0, (int)h0, 0xB1, 0xF, 0xF, false);
        const unsigned g1 = (unsigned)__builtin_amdgcn_update_dpp(0, (int)h1, 0xB1, 0xF, 0xF, false);
        if (!(lane & 1) && ((hz >> (lane & 48)) & 0xFFFFull))
            st128(u32x4{h0, h1, g0, g1}, rh, lane * 8, blk * 512, 17);
    }
    st128(l, rd, lane * 16, blk * 1536, 17);
    const unsigned m0 = (unsigned)__builtin_amdgcn_update_dpp(0, (int)n0, 0xB1, 0xF, 0xF, false);
    const unsigned m1 = (unsigned)__builtin_amdgcn_update_dpp(0, (int)n1, 0xB1, 0xF, 0xF, false);
    if (!(lane & 1)) st128(u32x4{n0, n1, m0, m1}, rd, 1024 + lane * 8, blk * 1536, 17);
}
#define STAMPW(i, w) do {} while (0)
#define STAMP(i) STAMPW(i, 0)

__global__ void __launch_bounds__(NTHR) ag_gemm_kernel(Params p) {
    __shared__ __attribute__((aligned(16))) char smem[4 * 32768 + 512];
    const int rank = p.rank;
    char* me = p.ws[rank];
    const int tid = threadIdx.x, lane = tid & 63;
    const int wv = __builtin_amdgcn_readfirstlane(tid >> 6);
    const int bid = blockIdx.x;
    const int Q = rank >> 1;
    const unsigned xswz = (unsigned)((lane >> 3) * 128 + (((lane & 7) ^ (lane >> 3)) * 16));
    STAMP(0);
    const unsigned callv = __hip_atomic_load((const unsigned*)(me + CALLW), __ATOMIC_RELAXED, __HIP_MEMORY_SCOPE_AGENT);
    unsigned* tmo = (unsigned*)(me + TMOW);

    if (bid > 0 && bid < PB0) {
        const unsigned call = __builtin_amdgcn_readfirstlane(callv) + 1u;
        const __amdgpu_buffer_rsrc_t rz = mk_rsrc(me + H_OFF + (size_t)((call + 1u) & 1u) * H_BYTES, (unsigned)H_BYTES);
        for (unsigned o = (unsigned)((bid - 1) * NTHR + tid) * 16u; o < (unsigned)H_BYTES; o += (PB0 - 1) * NTHR * 16)
            st128(u32x4{0u, 0u, 0u, 0u}, rz, o, 0, 17);
        wait_vm<0>();
        bar();
        if (tid == 0)
            __hip_atomic_store((unsigned long long*)(me + ZFLAG + bid * 128), (unsigned long long)call << 32, __ATOMIC_RELAXED, __HIP_MEMORY_SCOPE_AGENT);
        return;
    }
    if (bid >= RB0) {
        typedef unsigned u32x2 __attribute__((ext_vector_type(2)));
        const unsigned call = __builtin_amdgcn_readfirstlane(callv) + 1u;
        const int e = (bid - RB0) % 7;
        const int src = e < 6 ? 2 * ((Q + 1 + (e >> 1)) & 3) + (e & 1) : rank ^ 1;
        const __amdgpu_buffer_rsrc_t rd = mk_rsrc(me + D_OFF, (unsigned)D_BYTES);
        const __amdgpu_buffer_rsrc_t rh = mk_rsrc(me + H_OFF + (size_t)(call & 1u) * H_BYTES, (unsigned)H_BYTES);
        const __amdgpu_buffer_rsrc_t ro = mk_rsrc(me + A_OFF, (unsigned)(A_BYTES + B_BYTES));
        const char* pf = me + PFLAG + (size_t)src * (NITEMS * 128);
        char* rf = me + IFLAG + (size_t)src * (NITEMS * 128);
        for (int t = NPL0 * NSL + ((bid - RB0) / 7) * NWV + wv; t < NPL * NSL; t += RWG * NWV) {
            const unsigned long long t0 = __builtin_amdgcn_s_memrealtime();
            unsigned fv;
            while (true) {
                fv = __builtin_amdgcn_readfirstlane(ld_sys_asm(pf + t * 128));
                if ((fv >> 14) == (call & 0x3FFFFu)) break;
                if (give_up(t0, tmo, call)) {
                    if (lane == 0) __hip_atomic_store(tmo, call, __ATOMIC_RELAXED, __HIP_MEMORY_SCOPE_AGENT);
                    return;
                }
                __builtin_amdgcn_s_sleep(2);
            }
            const unsigned Rx = (((fv >> 7) & 127u) | 0x80u) * 0x01010101u, Rw = ((fv & 127u) | 0x80u) * 0x01010101u;
            const int st = t >> 2, sl = t & 3;
            u32x4 l[5];
            u32x2 n[5], h[5];
            int blk[5];
#pragma unroll
            for (int i = 0; i < 5; ++i) {
                blk[i] = e == 6 ? 20 * (src & 1) + 5 * sl + i : i == 0 ? 40 + 4 * e + sl : 64 + ((2 + e) * 4 + i - 1) * 4 + sl;
                const int b = st * BPS + blk[i];
                l[i] = __builtin_amdgcn_raw_buffer_load_b128(rd, lane * 16, b * 1536, 17);
                n[i] = __builtin_amdgcn_raw_buffer_load_b64(rd, 1024 + lane * 8, b * 1536, 17);
                h[i] = __builtin_amdgcn_raw_buffer_load_b64(rh, lane * 8, b * 512, 17);
            }
#pragma unroll
            for (int i = 0; i < 5; ++i) {
                const bool isx = blk[i] < 64;
                const unsigned R = isx ? Rx : Rw;
                const int wb = blk[i] - 64;
                const int so = isx ? (st * AROWS + blk[i] * 16) * (BK * 2)
                                   : (int)A_BYTES + (st * (8 * BCOLS) + (wb >> 2) * 512 + (wb & 3) * 64) * 16;
                const unsigned vo = isx ? xswz : (unsigned)lane * 16u;
                st128(dec8(l[i].x, l[i].y, n[i].x, h[i].x, R), ro, vo, so, 17);
                st128(dec8(l[i].z, l[i].w, n[i].y, h[i].y, R), ro, vo, so + (isx ? 1024 : 4096), 17);
            }
            wait_vm<0>();
            if (lane == 0) st_sys(rf + t * 128, call);
        }
        return;
    }

    if (bid == 0) {
        if (wv >= 3) return;
        const unsigned call = __builtin_amdgcn_readfirstlane(callv) + 1u;
        volatile int* ctl = (volatile int*)(smem + 4 * 32768);
        char* gws = p.ws[0];
#pragma unroll
        for (int i = 1; i < NR; ++i) gws = (lane & 7) == i ? p.ws[i] : gws;
        unsigned long long* gdst = (unsigned long long*)(gws + GGRAN + ((call & 1u) * NR + rank) * 128);
        bool bad = false;
        {
            const unsigned long long* lg = (const unsigned long long*)(wv < 2 ? me + LGRAN + tid * 128 : me + ZFLAG + (1 + lane % (PB0 - 1)) * 128);
            const unsigned long long t0 = __builtin_amdgcn_s_memrealtime();
            unsigned long long v;
            while (true) {
                v = __hip_atomic_load(lg, __ATOMIC_RELAXED, __HIP_MEMORY_SCOPE_AGENT);
                if (__all((unsigned)(v >> 32) == call)) break;
                if (give_up(t0, tmo, call)) { bad = true; break; }
                __builtin_amdgcn_s_sleep(1);
            }
            unsigned am = wave_umax((unsigned)v);
            if (lane == 0) lds_st(ctl + wv, (int)am);
            __syncthreads();
            am = umax((unsigned)lds_ld(ctl), (unsigned)lds_ld(ctl + 1));
            if (tid < NR)
                __hip_atomic_store(gdst, ((unsigned long long)call << 32) | am, __ATOMIC_RELAXED, __HIP_MEMORY_SCOPE_SYSTEM);
        }
        STAMP(1);
        {
            const bool need = tid < 128 && (tid >> 5) != Q;
            const unsigned* df = (const unsigned*)(me + DFLAG + (tid & 127) * 128);
            const unsigned long long t0 = __builtin_amdgcn_s_memrealtime();
            while (true) {
                const unsigned v = __hip_atomic_load(df, __ATOMIC_RELAXED, __HIP_MEMORY_SCOPE_SYSTEM);
                if (__all(!need || v >= call)) break;
                if (give_up(t0, tmo, call)) { bad = true; break; }
                __builtin_amdgcn_s_sleep(1);
            }
            if (bad && lane == 0) {
                __hip_atomic_store(tmo, call, __ATOMIC_RELAXED, __HIP_MEMORY_SCOPE_AGENT);
                p.out[rank][(size_t)(2 * Q) * MS * NS] = 0x7F;
            }
            __syncthreads();
            if (tid == 0)
                __hip_atomic_store((unsigned*)(me + CALLW), call, __ATOMIC_RELAXED, __HIP_MEMORY_SCOPE_AGENT);
            if (tid < NSTREAM)
                __hip_atomic_store((unsigned*)(me + TICKW + tid * 128), 0u, __ATOMIC_RELAXED, __HIP_MEMORY_SCOPE_AGENT);
        }
        STAMP(2);
        return;
    }

    if (bid < CB0) {
        const int stream = (bid - PB0) >> 3, j0 = bid & 7;
        if (wv >= 7 || (stream >= 8 && wv == 0)) return;
        volatile int* tk = (volatile int*)smem;
        const unsigned tk_a = (unsigned)(uintptr_t)(__attribute__((address_space(3))) char*)smem;
        int dst = rank, fidx = rank;
        bool isx = true;
        int xrow = 16 * wv, arow = 320 * (rank & 1) + 16 * wv, xrs = 80;
        int wcol = 0, slot = 0;
        if (stream < 6) {
            const int i = stream >> 1, m = stream & 1;
            const int sl = 2 * ((2 - i) & 3) + (rank & 1);
            dst = 2 * ((Q + 1 + i) & 3) + m;
            isx = wv == 0;
            xrow = 320 + 64 * i; arow = 640 + 64 * sl; xrs = 16;
            wcol = 256 * m; slot = 2 + sl;
        } else if (stream == 6) {
            dst = rank ^ 1;
        } else if (stream >= 8) {
            isx = false; fidx = stream; wcol = 256 * (stream - 8); slot = stream - 8;
        }
        char* dws = p.ws[dst & 7];
        const float* xsmp = p.x + (size_t)(8 * lane) * KDIM + ((lane * 61) & 1023) * 4;
        const float* wsmp = p.w + (size_t)(64 * lane) * NS + ((lane * 7) & 127) * 4;
        const unsigned npl = stream < 7 ? (NPL - NPL0) * NSL : 0;
        auto planes = [&](int t) { return (unsigned)(t - NPL0 * NSL) < npl; };
        if (wv == 6) {
            unsigned* tc = (unsigned*)(me + TICKW + stream * 128);
            int t1 = j0 + 8;
            for (int i = 0;; ++i) {
                int v = NITEMS;
                if (t1 < NITEMS) {
                    if (lane == 0) v = (int)__hip_atomic_fetch_add(tc, 1u, __ATOMIC_RELAXED, __HIP_MEMORY_SCOPE_AGENT) + 16;
                    v = __builtin_amdgcn_readfirstlane(v);
                    if ((unsigned)v > (unsigned)NITEMS) v = NITEMS;
                    if (lane == 0) lds_st(tk + ((i + 2) & 7), v);
                    asm volatile("s_waitcnt lgkmcnt(0)" ::: "memory");
                }
                bar();
                if (t1 >= NITEMS) break;
                t1 = v;
            }
            bar();
            return;
        }
        if (wv == 5) {
            const unsigned call = __builtin_amdgcn_readfirstlane(callv) + 1u;
            const unsigned pcall = (call << 14) | (exp_ref(xsmp) << 7) | exp_ref(wsmp);
            char* fb = dws + IFLAG + (size_t)fidx * (NITEMS * 128);
            char* pb = dws + PFLAG + (size_t)rank * (NITEMS * 128);
            auto publish = [&](int t) { if (lane == 0) st_sys((planes(t) ? pb : fb) + t * 128, planes(t) ? pcall : call); };
            int tp = -1, t0 = j0, t1 = j0 + 8;
            for (int i = 0;; ++i) {
                bar();
                if (tp >= 0) publish(tp);
                if (t1 >= NITEMS) break;
                tp = t0; t0 = t1;
                t1 = __builtin_amdgcn_readfirstlane(lds_ld_asm(tk_a + ((i + 2) & 7) * 4));
            }
            bar();
            publish(t0);
            return;
        }
        const __amdgpu_buffer_rsrc_t rd = mk_rsrc(dws + D_OFF, (unsigned)D_BYTES);
        __amdgpu_buffer_rsrc_t rh;
        unsigned R4;
        auto setup = [&](const float* smp) {
            R4 = (exp_ref(smp) | 0x80u) * 0x01010101u;
            rh = mk_rsrc(dws + H_OFF + (size_t)((__builtin_amdgcn_readfirstlane(callv) + 1u) & 1u) * H_BYTES, (unsigned)H_BYTES);
        };
        if (isx) {
            const __amdgpu_buffer_rsrc_t r0 = mk_rsrc(dws + A_OFF, (unsigned)A_BYTES);
            const unsigned vo0 = (unsigned)((xrow + (lane >> 3)) * KDIM + (lane & 7) * 8) * 4u, vo1 = vo0 + 8 * KDIM * 4;
            const unsigned doff = (unsigned)(arow * BK * 2) + xswz;
            f32x4 va[4], vb[4];
            auto ld = [&](f32x4 (&v)[4], int t) {
                const float* sp = p.x + (size_t)((t & 3) * xrs) * KDIM + (t >> 2) * BK;
                asm volatile("s_nop 4\n\t"
                             "global_load_dwordx4 %0, %4, %6\n\tglobal_load_dwordx4 %1, %4, %6 offset:16\n\t"
                             "global_load_dwordx4 %2, %5, %6\n\tglobal_load_dwordx4 %3, %5, %6 offset:16"
                             : "=&v"(v[0]), "=&v"(v[1]), "=&v"(v[2]), "=&v"(v[3]) : "v"(vo0), "v"(vo1), "s"(sp) : "memory");
            };
            auto step = [&](f32x4 (&cur)[4], f32x4 (&nxt)[4], int t, int tn) {
                ld(nxt, tn < NITEMS ? tn : t);
                asm volatile("s_waitcnt vmcnt(6)" : "+v"(cur[0]), "+v"(cur[1]), "+v"(cur[2]), "+v"(cur[3]) :: "memory");
                const unsigned ua[4] = {pk_bf16(cur[0].x, cur[0].y), pk_bf16(cur[0].z, cur[0].w), pk_bf16(cur[1].x, cur[1].y), pk_bf16(cur[1].z, cur[1].w)};
                const unsigned ub[4] = {pk_bf16(cur[2].x, cur[2].y), pk_bf16(cur[2].z, cur[2].w), pk_bf16(cur[3].x, cur[3].y), pk_bf16(cur[3].z, cur[3].w)};
                if (planes(t)) {
                    put_block(ua, ub, R4, lane, rd, rh, (t >> 2) * BPS + ((arow + (t & 3) * xrs) >> 4));
                } else {
                    const int so = ((t >> 2) * AROWS + (t & 3) * xrs) * (BK * 2);
                    st128(u32x4{ua[0], ua[1], ua[2], ua[3]}, r0, doff, so, 17);
                    st128(u32x4{ub[0], ub[1], ub[2], ub[3]}, r0, doff + 1024, so, 17);
                }
                wait_vm<6>();
                bar();
            };
            int t = j0, tn = j0 + 8;
            ld(va, t);
            setup(xsmp);
            asm volatile("s_waitcnt vmcnt(0)" : "+v"(va[0]), "+v"(va[1]), "+v"(va[2]), "+v"(va[3]) :: "memory");
            for (int i = 0;; i += 2) {
                step(va, vb, t, tn);
                if (tn >= NITEMS) break;
                t = tn; tn = __builtin_amdgcn_readfirstlane(lds_ld_asm(tk_a + ((i + 2) & 7) * 4));
                step(vb, va, t, tn);
                if (tn >= NITEMS) break;
                t = tn; tn = __builtin_amdgcn_readfirstlane(lds_ld_asm(tk_a + ((i + 3) & 7) * 4));
            }
            asm volatile("s_waitcnt vmcnt(0)" : "+v"(va[0]), "+v"(va[1]), "+v"(va[2]), "+v"(va[3]),
                                                "+v"(vb[0]), "+v"(vb[1]), "+v"(vb[2]), "+v"(vb[3]) :: "memory");
        } else {
            const __amdgpu_buffer_rsrc_t r0 = mk_rsrc(dws + B_OFF, (unsigned)B_BYTES);
            const int col = ((lane >> 2) & 3) * 16 + (lane >> 4) * 4 + (lane & 3);
            const unsigned vo0 = (unsigned)(((wv - 1) * 16 + 2) * NS + wcol + col) * 4u;
            const unsigned vo1 = vo0 + 4 * NS * 4, vo2 = vo0 + 8 * NS * 4, vo3 = vo0 + 12 * NS * 4;
            const unsigned doff = (unsigned)((slot * 8 + (wv - 1) * 2) * 256 + lane) * 16u;
            float va[16], vb[16];
#define AGK_V16(c, v) c(v[0]), c(v[1]), c(v[2]), c(v[3]), c(v[4]), c(v[5]), c(v[6]), c(v[7]), \
                      c(v[8]), c(v[9]), c(v[10]), c(v[11]), c(v[12]), c(v[13]), c(v[14]), c(v[15])
            auto ld = [&](float (&v)[16], int t) {
                const float* sp = p.w + (size_t)((t >> 2) * BK) * NS + (t & 3) * 64;
                asm volatile("s_nop 4\n\t"
                             "global_load_dword %0, %16, %20 offset:-4096\n\tglobal_load_dword %1, %16, %20 offset:-2048\n\t"
                             "global_load_dword %2, %16, %20\n\tglobal_load_dword %3, %16, %20 offset:2048\n\t"
                             "global_load_dword %4, %17, %20 offset:-4096\n\tglobal_load_dword %5, %17, %20 offset:-2048\n\t"
                             "global_load_dword %6, %17, %20\n\tglobal_load_dword %7, %17, %20 offset:2048\n\t"
                             "global_load_dword %8, %18, %20 offset:-4096\n\tglobal_load_dword %9, %18, %20 offset:-2048\n\t"
                             "global_load_dword %10, %18, %20\n\tglobal_load_dword %11, %18, %20 offset:2048\n\t"
                             "global_load_dword %12, %19, %20 offset:-4096\n\tglobal_load_dword %13, %19, %20 offset:-2048\n\t"
                             "global_load_dword %14, %19, %20\n\tglobal_load_dword %15, %19, %20 offset:2048"
                             : AGK_V16("=&v", v) : "v"(vo0), "v"(vo1), "v"(vo2), "v"(vo3), "s"(sp) : "memory");
            };
            auto step = [&](float (&cur)[16], float (&nxt)[16], int t, int tn) {
                ld(nxt, tn < NITEMS ? tn : t);
                asm volatile("s_waitcnt vmcnt(18)" : AGK_V16("+v", cur) :: "memory");
                const unsigned ua[4] = {pk_bf16(cur[0], cur[1]), pk_bf16(cur[2], cur[3]), pk_bf16(cur[4], cur[5]), pk_bf16(cur[6], cur[7])};
                const unsigned ub[4] = {pk_bf16(cur[8], cur[9]), pk_bf16(cur[10], cur[11]), pk_bf16(cur[12], cur[13]), pk_bf16(cur[14], cur[15])};
                if (planes(t)) {
                    put_block(ua, ub, R4, lane, rd, rh, (t >> 2) * BPS + 64 + (slot * 4 + (wv - 1)) * 4 + (t & 3));
                } else {
                    const int so = ((t >> 2) * (8 * BCOLS) + (t & 3) * 64) * 16;
                    st128(u32x4{ua[0], ua[1], ua[2], ua[3]}, r0, doff, so, 17);
                    st128(u32x4{ub[0], ub[1], ub[2], ub[3]}, r0, doff + 256 * 16, so, 17);
                }
                wait_vm<18>();
                bar();
            };
            int t = j0, tn = j0 + 8;
            ld(va, t);
            setup(wsmp);
            asm volatile("s_waitcnt vmcnt(0)" : AGK_V16("+v", va) :: "memory");
            for (int i = 0;; i += 2) {
                step(va, vb, t, tn);
                if (tn >= NITEMS) break;
                t = tn; tn = __builtin_amdgcn_readfirstlane(lds_ld_asm(tk_a + ((i + 2) & 7) * 4));
                step(vb, va, t, tn);
                if (tn >= NITEMS) break;
                t = tn; tn = __builtin_amdgcn_readfirstlane(lds_ld_asm(tk_a + ((i + 3) & 7) * 4));
            }
            asm volatile("s_waitcnt vmcnt(0)" : AGK_V16("+v", va) :: "memory");
            asm volatile("" : AGK_V16("+v", vb) :: "memory");
        }
        wait_vm<0>();
        bar();
        STAMPW(1, 1);
        return;
    }

    const int cid = bid - CB0;
    const int xg = cid & 7, xi = cid >> 3;
    const int rb = (xg & 1) * 4 + (xi & 3);
    const int cb = (xg >> 1) * 4 + (xi >> 2);
    volatile int* ctl = (volatile int*)(smem + 4 * 32768);
    const unsigned ctl_a = (unsigned)(uintptr_t)(__attribute__((address_space(3))) char*)(smem + 4 * 32768);
    if (tid < 84) lds_st(ctl + tid, tid < 16 ? 0 : -1);
    __syncthreads();
    const unsigned call = __builtin_amdgcn_readfirstlane(callv) + 1u;
    auto grp_src = [&](int g) { return g < 5 ? 2 * Q : g < 10 ? 2 * Q + 1 : 2 * ((Q + 1 + ((g - 10) >> 1)) & 3) + ((g - 10) & 1); };
    const int csrc = (cb >> 1) < 2 ? 8 + (cb >> 1) : 2 * ((Q + 1 + (((cb >> 1) - 2) >> 1)) & 3) + (((cb >> 1) - 2) & 1);

    if (wv >= 8) {
        const int psel = (lane >> 2) & 3;
        const int psrc = psel == 0 ? grp_src(2 * rb) : psel == 1 ? grp_src(2 * rb + 1) : csrc;
        const bool dup = psel == 3 || (psel == 1 && grp_src(2 * rb) == grp_src(2 * rb + 1));
        const char* pfb = me + IFLAG + (size_t)psrc * (NITEMS * 128) + (lane & 3) * 128;
        int cur = (lane >> 4) * NPOLL + (wv - 8);
        bool seen = dup;
        bool bad = false;
        unsigned long long t0 = __builtin_amdgcn_s_memrealtime();
        auto push = [&](int g) {
            const int st = __builtin_amdgcn_readlane(cur, 16 * g);
            int pos = 0;
            if (lane == 0) {
                pos = __hip_atomic_fetch_add((int*)ctl, 1, __ATOMIC_RELAXED, __HIP_MEMORY_SCOPE_WORKGROUP);
                lds_st(ctl + 16 + pos, st);
            }
            if ((lane >> 4) == g) { cur += 4 * NPOLL; seen = dup; }
        };
        while (__any(cur < NST)) {
            if (!seen && cur < NST) seen = ld_sys_asm(pfb + cur * (NSL * 128)) == call;
            const unsigned long long ok = __ballot(seen && cur < NST);
            bool any = false;
#pragma unroll
            for (int g = 0; g < 4; ++g)
                if (((unsigned)(ok >> (16 * g)) & 0xFFFFu) == 0xFFFFu) { push(g); any = true; }
            if (any) continue;
            if (give_up(t0, tmo, call)) {
                bad = true;
#pragma unroll
                for (int g = 0; g < 4; ++g)
                    while (__builtin_amdgcn_readlane(cur, 16 * g) < NST) push(g);
                break;
            }
            __builtin_amdgcn_s_sleep(1);
        }
        if (wv > 8) {
            if (bad && lane == 0) {
                __hip_atomic_store(tmo, call, __ATOMIC_RELAXED, __HIP_MEMORY_SCOPE_AGENT);
                lds_st(ctl + 1, 1);
            }
            asm volatile("s_waitcnt lgkmcnt(0)" ::: "memory");
            return;
        }
        const unsigned long long* gp = (const unsigned long long*)(me + GGRAN + ((call & 1u) * NR + (lane & 7)) * 128);
        t0 = __builtin_amdgcn_s_memrealtime();
        unsigned long long v;
        while (true) {
            v = __hip_atomic_load(gp, __ATOMIC_RELAXED, __HIP_MEMORY_SCOPE_SYSTEM);
            if (__all((unsigned)(v >> 32) == call)) break;
            if (give_up(t0, tmo, call)) { bad = true; break; }
            __builtin_amdgcn_s_sleep(1);
        }
        const unsigned am = wave_umax((unsigned)v);
        if (lane == 0) {
            if (bad) {
                __hip_atomic_store(tmo, call, __ATOMIC_RELAXED, __HIP_MEMORY_SCOPE_AGENT);
                lds_st(ctl + 1, 1);
            }
            lds_st(ctl + 12, (int)am);
        }
        asm volatile("s_waitcnt lgkmcnt(0)" ::: "memory");
        return;
    }

    if (wv >= 4) {
        const int lw = wv - 4;
        const char* xsrc = me + A_OFF + (size_t)(rb * 128 + lw * 32) * (BK * 2) + lane * 16;
        const char* wsrc = me + B_OFF + (size_t)(((cb >> 1) * 8 + lw * 2) * 256 + (cb & 1) * 128 + lane) * 16;
        int issued = 0, landed = 0;
        auto retire = [&]() {
            const int out = issued - landed;
            if (out == 1) wait_vm<0>(); else if (out == 2) wait_vm<8>(); else if (out == 3) wait_vm<16>(); else wait_vm<24>();
            ++landed;
            lds_st_asm(ctl_a + 16 + lw * 4, landed);
        };
        while (issued < NST) {
            u32x4 fw; unsigned long long qe;
            asm volatile("ds_read_b128 %0, %2 offset:32\n\tds_read2_b32 %1, %3 offset0:0 offset1:1\n\ts_waitcnt lgkmcnt(0)"
                         : "=&v"(fw), "=&v"(qe) : "v"(ctl_a), "v"(ctl_a + 64 + issued * 4) : "memory");
            const unsigned fa = fw.x < fw.y ? fw.x : fw.y, fb = fw.z < fw.w ? fw.z : fw.w;
            const int fr = __builtin_amdgcn_readfirstlane((int)(fa < fb ? fa : fb)) + 4;
            const int q0 = __builtin_amdgcn_readfirstlane((int)(unsigned)qe), q1 = __builtin_amdgcn_readfirstlane((int)(unsigned)(qe >> 32));
            const int rdy = q0 < 0 ? issued : q1 < 0 ? issued + 1 : issued + 2;
            const int lim = rdy < fr ? rdy : fr;
            bool progress = false;
#pragma unroll
            for (int k = 0; k < 2; ++k) {
                if (issued < lim) {
                    const int st = (k == 0 ? q0 : q1) & (NST - 1);
                    char* xb = smem + (issued & 3) * 32768 + lw * 4096;
                    char* wb = xb + 16384;
                    const char* xs = xsrc + (size_t)st * (AROWS * BK * 2);
                    const char* wsn = wsrc + (size_t)st * (8 * BCOLS * 16);
#pragma unroll
                    for (int i = 0; i < 4; ++i)
                        __builtin_amdgcn_global_load_lds((const unsigned*)(xs + i * 1024), (unsigned*)(xb + i * 1024), 16, 0, 17);
#pragma unroll
                    for (int i = 0; i < 4; ++i)
                        __builtin_amdgcn_global_load_lds((const unsigned*)(wsn + (i >> 1) * (256 * 16) + (i & 1) * 1024), (unsigned*)(wb + i * 1024), 16, 0, 17);
                    ++issued;
                    progress = true;
                }
                while (issued > landed && (issued - landed == 4 || vm_outstanding() <= 8 * (issued - landed - 1))) { retire(); progress = true; }
            }
            if (!progress) __builtin_amdgcn_s_sleep(1);
        }
        while (landed < NST) retire();
        return;
    }

    const int wm = wv >> 1, wn = wv & 1;
    const int ml = lane & 15, q = lane >> 4;
    f32x4 acc[4][4];
#pragma unroll
    for (int u = 0; u < 4; ++u)
#pragma unroll
        for (int t = 0; t < 4; ++t) acc[u][t] = f32x4{0.f, 0.f, 0.f, 0.f};
    const int xrd = (wm * 64 + ml) * 128;
    const int wrd = 16384 + (q * 128 + wn * 64 + ml) * 16;
    STAMP(1);
    bf16x8 wf0[4], xf0[4], wf1[4], xf1[4];
#pragma unroll
    for (int t = 0; t < 4; ++t) { wf1[t] = bf16x8{}; xf1[t] = bf16x8{}; }
    const int xsw0 = (q ^ (ml & 7)) * 16, xsw1 = ((4 + q) ^ (ml & 7)) * 16;
    int full = 0;
    for (int s = 0; s < NST; ++s) {
        while (full <= s) {
            full = lds_min4_asm(ctl_a + 16);
            if (full <= s) __builtin_amdgcn_s_sleep(1);
        }
        const char* sb = smem + (s & 3) * 32768;
#pragma unroll
        for (int t = 0; t < 4; ++t) wf0[t] = *(const bf16x8*)(sb + wrd + t * 256);
#pragma unroll
        for (int u = 0; u < 4; ++u) xf0[u] = *(const bf16x8*)(sb + xrd + u * 2048 + xsw0);
        __builtin_amdgcn_sched_barrier(0);
#pragma unroll
        for (int u = 0; u < 4; ++u)
#pragma unroll
            for (int t = 0; t < 4; ++t)
                acc[u][t] = __builtin_amdgcn_mfma_f32_16x16x32_bf16(wf1[t], xf1[u], acc[u][t], 0, 0, 0);
        __builtin_amdgcn_sched_barrier(0);
        u32x4 ln;
        asm volatile("ds_read_b128 %0, %1" : "=&v"(ln) : "v"(ctl_a + 16) : "memory");
#pragma unroll
        for (int t = 0; t < 4; ++t) wf1[t] = *(const bf16x8*)(sb + wrd + (4 * 128 * 16) + t * 256);
#pragma unroll
        for (int u = 0; u < 4; ++u) xf1[u] = *(const bf16x8*)(sb + xrd + u * 2048 + xsw1);
        __builtin_amdgcn_sched_barrier(0);
#pragma unroll
        for (int u = 0; u < 2; ++u)
#pragma unroll
            for (int t = 0; t < 4; ++t)
                acc[u][t] = __builtin_amdgcn_mfma_f32_16x16x32_bf16(wf0[t], xf0[u], acc[u][t], 0, 0, 0);
        __builtin_amdgcn_sched_barrier(0);
        asm volatile("s_waitcnt lgkmcnt(0)" : "+v"(ln) :: "memory");
        lds_st_asm(ctl_a + 32 + wv * 4, s + 1);
        {
            const unsigned a = ln.x < ln.y ? ln.x : ln.y, b = ln.z < ln.w ? ln.z : ln.w;
            full = __builtin_amdgcn_readfirstlane((int)(a < b ? a : b));
        }
        __builtin_amdgcn_sched_barrier(0);
#pragma unroll
        for (int u = 2; u < 4; ++u)
#pragma unroll
            for (int t = 0; t < 4; ++t)
                acc[u][t] = __builtin_amdgcn_mfma_f32_16x16x32_bf16(wf0[t], xf0[u], acc[u][t], 0, 0, 0);
        __builtin_amdgcn_sched_barrier(0);
    }
#pragma unroll
    for (int u = 0; u < 4; ++u)
#pragma unroll
        for (int t = 0; t < 4; ++t)
            acc[u][t] = __builtin_amdgcn_mfma_f32_16x16x32_bf16(wf1[t], xf1[u], acc[u][t], 0, 0, 0);
    STAMP(3);

    float mx = 0.f;
#pragma unroll
    for (int u = 0; u < 4; ++u)
#pragma unroll
        for (int t = 0; t < 4; ++t)
            mx = fmaxf(fmaxf(mx, fmaxf(acc[u][t][0], acc[u][t][1])), fmaxf(acc[u][t][2], acc[u][t][3]));
    {
        const unsigned wmx = wave_umax(__float_as_uint(mx));
        int tk = 0;
        if (lane == 0) {
            __hip_atomic_fetch_max((unsigned*)(ctl + 2), wmx, __ATOMIC_RELAXED, __HIP_MEMORY_SCOPE_WORKGROUP);
            tk = __hip_atomic_fetch_add((int*)(ctl + 3), 1, __ATOMIC_RELAXED, __HIP_MEMORY_SCOPE_WORKGROUP);
        }
        tk = __builtin_amdgcn_readfirstlane(tk);
        if (tk == 3) {
            const unsigned tm = (unsigned)lds_ld(ctl + 2);
            if (lane == 0)
                __hip_atomic_store((unsigned long long*)(me + LGRAN + cid * 128), ((unsigned long long)call << 32) | tm,
                                   __ATOMIC_RELAXED, __HIP_MEMORY_SCOPE_AGENT);
        }
    }
    const int g = rb * 2 + wm;
    const int srk = grp_src(g);
    const int row0 = g < 5 ? 64 * g : g < 10 ? 64 * (g - 5) : 320 + 64 * ((2 - ((g - 10) >> 1)) & 3);
    const int owner = ((cb >> 1) < 2 ? rank : csrc) & 7;
    const int coff = (cb >> 1) < 2 ? cb * 128 : 256 * (rank & 1) + (cb & 1) * 128;
    const __amdgpu_buffer_rsrc_t ro = mk_rsrc(p.out[owner], (unsigned)(KDIM * NS));
    const unsigned ooff = (unsigned)((srk * MS + row0 + ml) * NS + coff + wn * 64 + q * 16);
    unsigned* dfl = (unsigned*)(p.ws[owner] + DFLAG + (rank * 16 + rb * 2 + (cb & 1)) * 128);
    bar();
    STAMP(4);
    mx = __int_as_float(lds_ld(ctl + 12));
    const bool bad = lds_ld(ctl + 1) != 0;
    const float inv = mx > 0.f ? 448.0f / mx : 0.f;
#pragma unroll
    for (int u = 0; u < 4; ++u) {
        u32x4 o;
#pragma unroll
        for (int t = 0; t < 4; ++t) {
            const float a0 = __builtin_amdgcn_fmed3f(acc[u][t][0] * inv, 0.f, 448.f), a1 = __builtin_amdgcn_fmed3f(acc[u][t][1] * inv, 0.f, 448.f);
            const float a2 = __builtin_amdgcn_fmed3f(acc[u][t][2] * inv, 0.f, 448.f), a3 = __builtin_amdgcn_fmed3f(acc[u][t][3] * inv, 0.f, 448.f);
            int r = __builtin_amdgcn_cvt_pk_fp8_f32(a0, a1, 0, false);
            r = __builtin_amdgcn_cvt_pk_fp8_f32(a2, a3, r, true);
            o[t] = bad ? 0x7F7F7F7Fu : (unsigned)r;
        }
        st128(o, ro, ooff, u * (16 * NS), 17);
    }
    wait_vm<0>();
    bar();
    if (tid == 0 && owner != rank) st_sys(dfl, call);
    STAMP(5);
}

}

extern "C" void kernel_launch(int rank, int nranks, ncclComm_t comm,
                              hipStream_t stream,
                              void* const* d_in, const int* in_sizes, int n_in,
                              void* d_out, int out_size,
                              const int* in_dtypes, int out_dtype,
                              void* const* d_out_all,
                              void* const* d_ws_all, size_t ws_bytes) {
    (void)comm; (void)in_sizes; (void)n_in; (void)d_out; (void)out_size; (void)in_dtypes; (void)out_dtype;
    if (nranks != agk::NR || (unsigned)rank >= (unsigned)agk::NR || ws_bytes < agk::WS_NEED) {
        fprintf(stderr, "unsupported configuration\n");
        return;
    }
    agk::Params p;
    p.x = (const float*)d_in[0];
    p.w = (const float*)d_in[1];
    for (int i = 0; i < agk::NR; ++i) {
        p.ws[i] = (char*)d_ws_all[i];
        p.out[i] = (unsigned char*)d_out_all[i];
    }
    p.rank = rank;
    hipLaunchKernelGGL(agk::ag_gemm_kernel, dim3(agk::GRID), dim3(agk::NTHR), 0, stream, p);
    HIP_CHECK(hipGetLastError());
}
```

```cpp
#include <hip/hip_runtime.h>
#include <rccl/rccl.h>
#include <cstdio>
#include <cstdint>

#define HIP_CHECK(expr) do { hipError_t _e = (expr); if (_e != hipSuccess) { \
    fprintf(stderr, "HIP error %s at %s:%d\n", hipGetErrorString(_e), __FILE__, __LINE__); \
    return; } } while (0)

namespace agk {

constexpr int NR = 8;
constexpr int KDIM = 4096, MS = 512, NS = 512;
constexpr int BK = 64;
constexpr int NST = KDIM / BK;
constexpr int AROWS = 1024, BCOLS = 2048;
constexpr int NSTREAM = 10;
constexpr int NSL = 4;
constexpr int NITEMS = NST * NSL;
constexpr int NCONS = 128;
constexpr int NPL0 = 0, NPL = NST - 4;
constexpr int NPOLL = 2;
constexpr int NWV = 8 + NPOLL, NTHR = NWV * 64;
constexpr int RWG = 5;
constexpr int PB0 = 8, CB0 = PB0 + NSTREAM * 8, RB0 = CB0 + NCONS, GRID = RB0 + 7 * RWG;

constexpr size_t A_OFF = 0;
constexpr size_t A_BYTES = (size_t)AROWS * KDIM * 2;
constexpr size_t B_OFF = A_OFF + A_BYTES;
constexpr size_t B_BYTES = (size_t)BCOLS * KDIM * 2;
constexpr size_t C_OFF = B_OFF + B_BYTES;
constexpr size_t DFLAG = C_OFF;
constexpr size_t LGRAN = C_OFF + 16384;
constexpr size_t GGRAN = C_OFF + 32768;
constexpr size_t TICKW = C_OFF + 36864;
constexpr size_t CALLW = C_OFF + 40960;
constexpr size_t TMOW = C_OFF + 40960 + 128;
constexpr size_t TRACE = C_OFF + 65536;
constexpr size_t IFLAG = C_OFF + 262144;
constexpr size_t PFLAG = IFLAG + (size_t)NSTREAM * NITEMS * 128;
constexpr size_t WS_NEED = PFLAG + (size_t)NR * NITEMS * 128;
constexpr unsigned long long DEADLINE_TICKS = 50000000ull;
constexpr unsigned long long SLOW_TICKS = 20000ull;

typedef float f32x4 __attribute__((ext_vector_type(4)));
typedef float f32x2 __attribute__((ext_vector_type(2)));
typedef __bf16 bf16x2 __attribute__((ext_vector_type(2)));
typedef __bf16 bf16x8 __attribute__((ext_vector_type(8)));
typedef unsigned u32x4 __attribute__((ext_vector_type(4)));

struct Params {
    const float* x;
    const float* w;
    char* ws[NR];
    unsigned char* out[NR];
    int rank;
};

template <int N> __device__ __forceinline__ void wait_vm() {
    asm volatile("s_waitcnt vmcnt(%0)" ::"i"(N) : "memory");
}
__device__ __forceinline__ void bar() { asm volatile("s_barrier" ::: "memory"); }

__device__ __forceinline__ unsigned pk_bf16(float a, float b) {
    f32x2 v = {a, b};
    bf16x2 r = __builtin_convertvector(v, bf16x2);
    return __builtin_bit_cast(unsigned, r);
}
__device__ __forceinline__ unsigned ld_sys_asm(const void* p) {
    unsigned v;
    asm volatile("global_load_dword %0, %1, off sc0 sc1\n\ts_waitcnt vmcnt(0)" : "=&v"(v) : "v"(p) : "memory");
    return v;
}
__device__ __forceinline__ int lds_ld(volatile int* p) {
    return __hip_atomic_load((int*)p, __ATOMIC_RELAXED, __HIP_MEMORY_SCOPE_WORKGROUP);
}
__device__ __forceinline__ int lds_ld_asm(unsigned lds_addr) {
    int v;
    asm volatile("ds_read_b32 %0, %1\n\ts_waitcnt lgkmcnt(0)" : "=&v"(v) : "v"(lds_addr) : "memory");
    return v;
}
__device__ __forceinline__ int lds_min4_asm(unsigned lds_addr) {
    u32x4 v;
    asm volatile("ds_read_b128 %0, %1\n\ts_waitcnt lgkmcnt(0)" : "=&v"(v) : "v"(lds_addr) : "memory");
    const unsigned a = v.x < v.y ? v.x : v.y, b = v.z < v.w ? v.z : v.w;
    return __builtin_amdgcn_readfirstlane((int)(a < b ? a : b));
}
__device__ __forceinline__ void lds_st_asm(unsigned lds_addr, int v) {
    asm volatile("ds_write_b32 %0, %1" ::"v"(lds_addr), "v"(v) : "memory");
}
__device__ __forceinline__ int vm_outstanding() {
    unsigned r;
    asm volatile("s_getreg_b32 %0, hwreg(HW_REG_IB_STS)" : "=s"(r));
    return (int)((r & 15u) | ((r >> 18) & 48u));
}
__device__ __forceinline__ void lds_st(volatile int* p, int v) {
    __hip_atomic_store((int*)p, v, __ATOMIC_RELAXED, __HIP_MEMORY_SCOPE_WORKGROUP);
}
__device__ __forceinline__ void st_sys(void* p, unsigned v) {
    __hip_atomic_store((unsigned*)p, v, __ATOMIC_RELAXED, __HIP_MEMORY_SCOPE_SYSTEM);
}
__device__ __forceinline__ bool give_up(unsigned long long t0, unsigned* tmo, unsigned call) {
    const unsigned long long dt = __builtin_amdgcn_s_memrealtime() - t0;
    if (dt < SLOW_TICKS) return false;
    return dt > DEADLINE_TICKS || __builtin_amdgcn_readfirstlane(ld_sys_asm(tmo)) == call;
}
__device__ __forceinline__ __amdgpu_buffer_rsrc_t mk_rsrc(void* base, unsigned bytes) {
    return __builtin_amdgcn_make_buffer_rsrc(base, 0, bytes, 0x00020000);
}
__device__ __forceinline__ void st128(u32x4 v, __amdgpu_buffer_rsrc_t r, unsigned voff, int soff, int) {
    __builtin_amdgcn_raw_buffer_store_b128(v, r, voff, soff, 17);
    asm volatile("s_nop 1" ::"v"(v) : "memory");
}
__device__ __forceinline__ unsigned umax(unsigned a, unsigned b) { return a > b ? a : b; }
__device__ __forceinline__ unsigned wave_umax(unsigned v) {
    v = umax(v, (unsigned)__builtin_amdgcn_update_dpp(0, (int)v, 0xB1, 0xF, 0xF, false));
    v = umax(v, (unsigned)__builtin_amdgcn_update_dpp(0, (int)v, 0x4E, 0xF, 0xF, false));
    v = umax(v, (unsigned)__builtin_amdgcn_update_dpp(0, (int)v, 0x141, 0xF, 0xF, false));
    v = umax(v, (unsigned)__builtin_amdgcn_update_dpp(0, (int)v, 0x140, 0xF, 0xF, false));
    const unsigned a = __builtin_amdgcn_readlane(v, 0), b = __builtin_amdgcn_readlane(v, 16);
    const unsigned c = __builtin_amdgcn_readlane(v, 32), d = __builtin_amdgcn_readlane(v, 48);
    return umax(umax(a, b), umax(c, d));
}
__device__ __forceinline__ unsigned exp_ref(const float* sp) {
    const f32x4 v = *(const f32x4*)sp;
    const unsigned m = umax(umax(__float_as_uint(v.x) & 0x7FFFFFFFu, __float_as_uint(v.y) & 0x7FFFFFFFu),
                            umax(__float_as_uint(v.z) & 0x7FFFFFFFu, __float_as_uint(v.w) & 0x7FFFFFFFu));
    const unsigned r = ((wave_umax(m) >> 23) + 1u) >> 1;
    return r > 127u ? 127u : r;
}
__device__ __forceinline__ void enc8(const unsigned (&u)[4], unsigned R4, unsigned& l0, unsigned& l1, unsigned& n, unsigned& h) {
    const unsigned ha = __builtin_amdgcn_perm(u[1], u[0], 0x07050301u), hb = __builtin_amdgcn_perm(u[3], u[2], 0x07050301u);
    l0 = __builtin_amdgcn_perm(u[1], u[0], 0x06040200u);
    l1 = __builtin_amdgcn_perm(u[3], u[2], 0x06040200u);
    const unsigned da = (R4 - (ha & 0x7F7F7F7Fu)) & 0x7F7F7F7Fu, db = (R4 - (hb & 0x7F7F7F7Fu)) & 0x7F7F7F7Fu;
    n = (da & 0x07070707u) | ((ha >> 4) & 0x08080808u) | ((db & 0x07070707u) << 4) | (hb & 0x80808080u);
    h = ((da >> 3) & 0x0F0F0F0Fu) | ((db << 1) & 0xF0F0F0F0u);
}
__device__ __forceinline__ u32x4 dec8(unsigned l0, unsigned l1, unsigned n, unsigned h, unsigned R4) {
    const unsigned da = (n & 0x07070707u) | ((h & 0x0F0F0F0Fu) << 3);
    const unsigned db = ((n >> 4) & 0x07070707u) | ((h >> 1) & 0x78787878u);
    const unsigned ha = ((R4 - da) & 0x7F7F7F7Fu) | ((n << 4) & 0x80808080u);
    const unsigned hb = ((R4 - db) & 0x7F7F7F7Fu) | (n & 0x80808080u);
    return u32x4{__builtin_amdgcn_perm(ha, l0, 0x05010400u), __builtin_amdgcn_perm(ha, l0, 0x07030602u),
                 __builtin_amdgcn_perm(hb, l1, 0x05010400u), __builtin_amdgcn_perm(hb, l1, 0x07030602u)};
}
__device__ __forceinline__ unsigned put_block(const unsigned (&ua)[4], const unsigned (&ub)[4], unsigned R4, int lane,
                                              __amdgpu_buffer_rsrc_t r, int so, int p2) {
    u32x4 l; unsigned n0, n1, h0, h1, la, lb;
    enc8(ua, R4, la, lb, n0, h0); l.x = la; l.y = lb;
    enc8(ub, R4, la, lb, n1, h1); l.z = la; l.w = lb;
    const unsigned long long hz = __ballot((h0 | h1) != 0u);
    unsigned hm = 0;
    if (hz) {
        hm = ((hz & 0xFFFFull) ? 1u : 0u) | ((hz & 0xFFFF0000ull) ? 2u : 0u) | ((hz & 0xFFFF00000000ull) ? 4u : 0u) | ((hz >> 48) ? 8u : 0u);
        const unsigned g0 = (unsigned)__builtin_amdgcn_update_dpp(0, (int)h0, 0xB1, 0xF, 0xF, false);
        const unsigned g1 = (unsigned)__builtin_amdgcn_update_dpp(0, (int)h1, 0xB1, 0xF, 0xF, false);
        if (!(lane & 1) && ((hm >> (lane >> 4)) & 1u)) st128(u32x4{h0, h1, g0, g1}, r, 512 + lane * 8, so + p2, 17);
    }
    st128(l, r, lane * 16, so, 17);
    const unsigned m0 = (unsigned)__builtin_amdgcn_update_dpp(0, (int)n0, 0xB1, 0xF, 0xF, false);
    const unsigned m1 = (unsigned)__builtin_amdgcn_update_dpp(0, (int)n1, 0xB1, 0xF, 0xF, false);
    if (!(lane & 1)) st128(u32x4{n0, n1, m0, m1}, r, lane * 8, so + p2, 17);
    return hm;
}
#define STAMPW(i, w) do {} while (0)
#define STAMP(i) STAMPW(i, 0)

__global__ void __launch_bounds__(NTHR) ag_gemm_kernel(Params p) {
    __shared__ __attribute__((aligned(16))) char smem[4 * 32768 + 512];
    const int rank = p.rank;
    char* me = p.ws[rank];
    const int tid = threadIdx.x, lane = tid & 63;
    const int wv = __builtin_amdgcn_readfirstlane(tid >> 6);
    const int bid = blockIdx.x;
    const int Q = rank >> 1;
    const unsigned xswz = (unsigned)((lane >> 3) * 128 + (((lane & 7) ^ (lane >> 3)) * 16));
    STAMP(0);
    const unsigned callv = __hip_atomic_load((const unsigned*)(me + CALLW), __ATOMIC_RELAXED, __HIP_MEMORY_SCOPE_AGENT);
    unsigned* tmo = (unsigned*)(me + TMOW);

    if (bid > 0 && bid < PB0) return;
    if (bid >= RB0) {
        typedef unsigned u32x2 __attribute__((ext_vector_type(2)));
        const unsigned call = __builtin_amdgcn_readfirstlane(callv) + 1u;
        const int e = (bid - RB0) % 7;
        const int src = e < 6 ? 2 * ((Q + 1 + (e >> 1)) & 3) + (e & 1) : rank ^ 1;
        const __amdgpu_buffer_rsrc_t ro = mk_rsrc(me + A_OFF, (unsigned)(A_BYTES + B_BYTES));
        const char* pf = me + PFLAG + (size_t)src * (NITEMS * 128);
        char* rf = me + IFLAG + (size_t)src * (NITEMS * 128);
        for (int t = NPL0 * NSL + ((bid - RB0) / 7) * NWV + wv; t < NPL * NSL; t += RWG * NWV) {
            const unsigned long long t0 = __builtin_amdgcn_s_memrealtime();
            unsigned fv, hm;
            while (true) {
                const unsigned long long g = __hip_atomic_load((const unsigned long long*)(pf + t * 128), __ATOMIC_RELAXED, __HIP_MEMORY_SCOPE_SYSTEM);
                fv = __builtin_amdgcn_readfirstlane((unsigned)g);
                hm = __builtin_amdgcn_readfirstlane((unsigned)(g >> 32));
                if ((fv >> 14) == (call & 0x3FFFFu)) break;
                if (give_up(t0, tmo, call)) {
                    if (lane == 0) __hip_atomic_store(tmo, call, __ATOMIC_RELAXED, __HIP_MEMORY_SCOPE_AGENT);
                    return;
                }
                __builtin_amdgcn_s_sleep(2);
            }
            const unsigned Rx = (((fv >> 7) & 127u) | 0x80u) * 0x01010101u, Rw = ((fv & 127u) | 0x80u) * 0x01010101u;
            const int st = t >> 2, sl = t & 3;
            u32x4 l[5];
            u32x2 n[5], h[5];
            int so[5];
#pragma unroll
            for (int i = 0; i < 5; ++i) {
                const bool isx = e == 6 || i == 0;
                so[i] = e == 6 ? (st * AROWS + 320 * (src & 1) + 80 * sl + 16 * i) * (BK * 2)
                      : i == 0 ? (st * AROWS + 640 + 64 * e + 16 * sl) * (BK * 2)
                               : (int)A_BYTES + (st * (8 * BCOLS) + ((2 + e) * 8 + (i - 1) * 2) * 256 + sl * 64) * 16;
                const int p2 = isx ? 1024 : 4096;
                l[i] = __builtin_amdgcn_raw_buffer_load_b128(ro, lane * 16, so[i], 17);
                n[i] = __builtin_amdgcn_raw_buffer_load_b64(ro, lane * 8, so[i] + p2, 17);
                h[i] = u32x2{0u, 0u};
                if ((hm >> (4 * i + (lane >> 4))) & 1u) h[i] = __builtin_amdgcn_raw_buffer_load_b64(ro, 512 + lane * 8, so[i] + p2, 17);
            }
#pragma unroll
            for (int i = 0; i < 5; ++i) {
                const bool isx = e == 6 || i == 0;
                const unsigned R = isx ? Rx : Rw;
                const unsigned vo = isx ? xswz : (unsigned)lane * 16u;
                const u32x4 u0 = dec8(l[i].x, l[i].y, n[i].x, h[i].x, R), u1 = dec8(l[i].z, l[i].w, n[i].y, h[i].y, R);
                st128(u0, ro, vo, so[i], 17);
                st128(u1, ro, vo, so[i] + (isx ? 1024 : 4096), 17);
            }
            wait_vm<0>();
            if (lane == 0) st_sys(rf + t * 128, call);
        }
        return;
    }

    if (bid == 0) {
        if (wv >= 2) return;
        const unsigned call = __builtin_amdgcn_readfirstlane(callv) + 1u;
        volatile int* ctl = (volatile int*)(smem + 4 * 32768);
        char* gws = p.ws[0];
#pragma unroll
        for (int i = 1; i < NR; ++i) gws = (lane & 7) == i ? p.ws[i] : gws;
        unsigned long long* gdst = (unsigned long long*)(gws + GGRAN + ((call & 1u) * NR + rank) * 128);
        bool bad = false;
        {
            const unsigned long long* lg = (const unsigned long long*)(me + LGRAN + tid * 128);
            const unsigned long long t0 = __builtin_amdgcn_s_memrealtime();
            unsigned long long v;
            while (true) {
                v = __hip_atomic_load(lg, __ATOMIC_RELAXED, __HIP_MEMORY_SCOPE_AGENT);
                if (__all((unsigned)(v >> 32) == call)) break;
                if (give_up(t0, tmo, call)) { bad = true; break; }
                __builtin_amdgcn_s_sleep(1);
            }
            unsigned am = wave_umax((unsigned)v);
            if (lane == 0) lds_st(ctl + wv, (int)am);
            __syncthreads();
            am = umax((unsigned)lds_ld(ctl), (unsigned)lds_ld(ctl + 1));
            if (tid < NR)
                __hip_atomic_store(gdst, ((unsigned long long)call << 32) | am, __ATOMIC_RELAXED, __HIP_MEMORY_SCOPE_SYSTEM);
        }
        STAMP(1);
        {
            const bool need = tid < 128 && (tid >> 5) != Q;
            const unsigned* df = (const unsigned*)(me + DFLAG + (tid & 127) * 128);
            const unsigned long long t0 = __builtin_amdgcn_s_memrealtime();
            while (true) {
                const unsigned v = __hip_atomic_load(df, __ATOMIC_RELAXED, __HIP_MEMORY_SCOPE_SYSTEM);
                if (__all(!need || v >= call)) break;
                if (give_up(t0, tmo, call)) { bad = true; break; }
                __builtin_amdgcn_s_sleep(1);
            }
            if (bad && lane == 0) {
                __hip_atomic_store(tmo, call, __ATOMIC_RELAXED, __HIP_MEMORY_SCOPE_AGENT);
                p.out[rank][(size_t)(2 * Q) * MS * NS] = 0x7F;
            }
            __syncthreads();
            if (tid == 0)
                __hip_atomic_store((unsigned*)(me + CALLW), call, __ATOMIC_RELAXED, __HIP_MEMORY_SCOPE_AGENT);
            if (tid < NSTREAM)
                __hip_atomic_store((unsigned*)(me + TICKW + tid * 128), 0u, __ATOMIC_RELAXED, __HIP_MEMORY_SCOPE_AGENT);
        }
        STAMP(2);
        return;
    }

    if (bid < CB0) {
        const int stream = (bid - PB0) >> 3, j0 = bid & 7;
        if (wv >= 7 || (stream >= 8 && wv == 0)) return;
        volatile int* tk = (volatile int*)smem;
        const unsigned tk_a = (unsigned)(uintptr_t)(__attribute__((address_space(3))) char*)smem;
        int dst = rank, fidx = rank;
        bool isx = true;
        int xrow = 16 * wv, arow = 320 * (rank & 1) + 16 * wv, xrs = 80;
        int wcol = 0, slot = 0;
        if (stream < 6) {
            const int i = stream >> 1, m = stream & 1;
            const int sl = 2 * ((2 - i) & 3) + (rank & 1);
            dst = 2 * ((Q + 1 + i) & 3) + m;
            isx = wv == 0;
            xrow = 320 + 64 * i; arow = 640 + 64 * sl; xrs = 16;
            wcol = 256 * m; slot = 2 + sl;
        } else if (stream == 6) {
            dst = rank ^ 1;
        } else if (stream >= 8) {
            isx = false; fidx = stream; wcol = 256 * (stream - 8); slot = stream - 8;
        }
        char* dws = p.ws[dst & 7];
        const float* xsmp = p.x + (size_t)(8 * lane) * KDIM + ((lane * 61) & 1023) * 4;
        const float* wsmp = p.w + (size_t)(64 * lane) * NS + ((lane * 7) & 127) * 4;
        const unsigned npl = stream < 7 ? (NPL - NPL0) * NSL : 0;
        auto planes = [&](int t) { return (unsigned)(t - NPL0 * NSL) < npl; };
        if (wv == 6) {
            unsigned* tc = (unsigned*)(me + TICKW + stream * 128);
            int t1 = j0 + 8;
            for (int i = 0;; ++i) {
                int v = NITEMS;
                if (t1 < NITEMS) {
                    if (lane == 0) v = (int)__hip_atomic_fetch_add(tc, 1u, __ATOMIC_RELAXED, __HIP_MEMORY_SCOPE_AGENT) + 16;
                    v = __builtin_amdgcn_readfirstlane(v);
                    if ((unsigned)v > (unsigned)NITEMS) v = NITEMS;
                    if (lane == 0) lds_st(tk + ((i + 2) & 7), v);
                    asm volatile("s_waitcnt lgkmcnt(0)" ::: "memory");
                }
                bar();
                if (t1 >= NITEMS) break;
                t1 = v;
            }
            bar();
            return;
        }
        if (wv == 5) {
            const unsigned call = __builtin_amdgcn_readfirstlane(callv) + 1u;
            const unsigned pcall = (call << 14) | (exp_ref(xsmp) << 7) | exp_ref(wsmp);
            char* fb = dws + IFLAG + (size_t)fidx * (NITEMS * 128);
            char* pb = dws + PFLAG + (size_t)rank * (NITEMS * 128);
            auto publish = [&](int t, int k) {
                if (planes(t)) {
                    const unsigned m = lane < 5 ? (unsigned)lds_ld_asm(tk_a + 64 + ((k & 7) * 8 + lane) * 4) << (4 * lane) : 0u;
                    const unsigned hm = __builtin_amdgcn_readlane(m, 0) | __builtin_amdgcn_readlane(m, 1) | __builtin_amdgcn_readlane(m, 2) |
                                        __builtin_amdgcn_readlane(m, 3) | __builtin_amdgcn_readlane(m, 4);
                    if (lane == 0)
                        __hip_atomic_store((unsigned long long*)(pb + t * 128), ((unsigned long long)hm << 32) | pcall,
                                           __ATOMIC_RELAXED, __HIP_MEMORY_SCOPE_SYSTEM);
                } else if (lane == 0) {
                    st_sys(fb + t * 128, call);
                }
            };
            int tp = -1, t0 = j0, t1 = j0 + 8, i = 0;
            for (;; ++i) {
                bar();
                if (tp >= 0) publish(tp, i - 1);
                if (t1 >= NITEMS) break;
                tp = t0; t0 = t1;
                t1 = __builtin_amdgcn_readfirstlane(lds_ld_asm(tk_a + ((i + 2) & 7) * 4));
            }
            bar();
            publish(t0, i);
            return;
        }
        unsigned R4;
        auto setup = [&](const float* smp) { R4 = (exp_ref(smp) | 0x80u) * 0x01010101u; };
        auto note = [&](unsigned hm, int k) {
            lds_st_asm(tk_a + 64 + ((k & 7) * 8 + wv) * 4, (int)hm);
            asm volatile("s_waitcnt lgkmcnt(0)" ::: "memory");
        };
        if (isx) {
            const __amdgpu_buffer_rsrc_t r0 = mk_rsrc(dws + A_OFF, (unsigned)A_BYTES);
            const unsigned vo0 = (unsigned)((xrow + (lane >> 3)) * KDIM + (lane & 7) * 8) * 4u, vo1 = vo0 + 8 * KDIM * 4;
            const unsigned doff = (unsigned)(arow * BK * 2) + xswz;
            f32x4 va[4], vb[4];
            auto ld = [&](f32x4 (&v)[4], int t) {
                const float* sp = p.x + (size_t)((t & 3) * xrs) * KDIM + (t >> 2) * BK;
                asm volatile("s_nop 4\n\t"
                             "global_load_dwordx4 %0, %4, %6\n\tglobal_load_dwordx4 %1, %4, %6 offset:16\n\t"
                             "global_load_dwordx4 %2, %5, %6\n\tglobal_load_dwordx4 %3, %5, %6 offset:16"
                             : "=&v"(v[0]), "=&v"(v[1]), "=&v"(v[2]), "=&v"(v[3]) : "v"(vo0), "v"(vo1), "s"(sp) : "memory");
            };
            auto step = [&](f32x4 (&cur)[4], f32x4 (&nxt)[4], int t, int tn, int k) {
                ld(nxt, tn < NITEMS ? tn : t);
                asm volatile("s_waitcnt vmcnt(6)" : "+v"(cur[0]), "+v"(cur[1]), "+v"(cur[2]), "+v"(cur[3]) :: "memory");
                const unsigned ua[4] = {pk_bf16(cur[0].x, cur[0].y), pk_bf16(cur[0].z, cur[0].w), pk_bf16(cur[1].x, cur[1].y), pk_bf16(cur[1].z, cur[1].w)};
                const unsigned ub[4] = {pk_bf16(cur[2].x, cur[2].y), pk_bf16(cur[2].z, cur[2].w), pk_bf16(cur[3].x, cur[3].y), pk_bf16(cur[3].z, cur[3].w)};
                const int so = ((t >> 2) * AROWS + (t & 3) * xrs) * (BK * 2);
                if (planes(t)) {
                    note(put_block(ua, ub, R4, lane, r0, so + arow * (BK * 2), 1024), k);
                } else {
                    st128(u32x4{ua[0], ua[1], ua[2], ua[3]}, r0, doff, so, 17);
                    st128(u32x4{ub[0], ub[1], ub[2], ub[3]}, r0, doff + 1024, so, 17);
                }
                wait_vm<6>();
                bar();
            };
            int t = j0, tn = j0 + 8;
            ld(va, t);
            setup(xsmp);
            asm volatile("s_waitcnt vmcnt(0)" : "+v"(va[0]), "+v"(va[1]), "+v"(va[2]), "+v"(va[3]) :: "memory");
            for (int i = 0;; i += 2) {
                step(va, vb, t, tn, i);
                if (tn >= NITEMS) break;
                t = tn; tn = __builtin_amdgcn_readfirstlane(lds_ld_asm(tk_a + ((i + 2) & 7) * 4));
                step(vb, va, t, tn, i + 1);
                if (tn >= NITEMS) break;
                t = tn; tn = __builtin_amdgcn_readfirstlane(lds_ld_asm(tk_a + ((i + 3) & 7) * 4));
            }
            asm volatile("s_waitcnt vmcnt(0)" : "+v"(va[0]), "+v"(va[1]), "+v"(va[2]), "+v"(va[3]),
                                                "+v"(vb[0]), "+v"(vb[1]), "+v"(vb[2]), "+v"(vb[3]) :: "memory");
        } else {
            const __amdgpu_buffer_rsrc_t r0 = mk_rsrc(dws + B_OFF, (unsigned)B_BYTES);
            const int col = ((lane >> 2) & 3) * 16 + (lane >> 4) * 4 + (lane & 3);
            const unsigned vo0 = (unsigned)(((wv - 1) * 16 + 2) * NS + wcol + col) * 4u;
            const unsigned vo1 = vo0 + 4 * NS * 4, vo2 = vo0 + 8 * NS * 4, vo3 = vo0 + 12 * NS * 4;
            const unsigned doff = (unsigned)((slot * 8 + (wv - 1) * 2) * 256 + lane) * 16u;
            float va[16], vb[16];
#define AGK_V16(c, v) c(v[0]), c(v[1]), c(v[2]), c(v[3]), c(v[4]), c(v[5]), c(v[6]), c(v[7]), \
                      c(v[8]), c(v[9]), c(v[10]), c(v[11]), c(v[12]), c(v[13]), c(v[14]), c(v[15])
            auto ld = [&](float (&v)[16], int t) {
                const float* sp = p.w + (size_t)((t >> 2) * BK) * NS + (t & 3) * 64;
                asm volatile("s_nop 4\n\t"
                             "global_load_dword %0, %16, %20 offset:-4096\n\tglobal_load_dword %1, %16, %20 offset:-2048\n\t"
                             "global_load_dword %2, %16, %20\n\tglobal_load_dword %3, %16, %20 offset:2048\n\t"
                             "global_load_dword %4, %17, %20 offset:-4096\n\tglobal_load_dword %5, %17, %20 offset:-2048\n\t"
                             "global_load_dword %6, %17, %20\n\tglobal_load_dword %7, %17, %20 offset:2048\n\t"
                             "global_load_dword %8, %18, %20 offset:-4096\n\tglobal_load_dword %9, %18, %20 offset:-2048\n\t"
                             "global_load_dword %10, %18, %20\n\tglobal_load_dword %11, %18, %20 offset:2048\n\t"
                             "global_load_dword %12, %19, %20 offset:-4096\n\tglobal_load_dword %13, %19, %20 offset:-2048\n\t"
                             "global_load_dword %14, %19, %20\n\tglobal_load_dword %15, %19, %20 offset:2048"
                             : AGK_V16("=&v", v) : "v"(vo0), "v"(vo1), "v"(vo2), "v"(vo3), "s"(sp) : "memory");
            };
            auto step = [&](float (&cur)[16], float (&nxt)[16], int t, int tn, int k) {
                ld(nxt, tn < NITEMS ? tn : t);
                asm volatile("s_waitcnt vmcnt(18)" : AGK_V16("+v", cur) :: "memory");
                const unsigned ua[4] = {pk_bf16(cur[0], cur[1]), pk_bf16(cur[2], cur[3]), pk_bf16(cur[4], cur[5]), pk_bf16(cur[6], cur[7])};
                const unsigned ub[4] = {pk_bf16(cur[8], cur[9]), pk_bf16(cur[10], cur[11]), pk_bf16(cur[12], cur[13]), pk_bf16(cur[14], cur[15])};
                const int so = ((t >> 2) * (8 * BCOLS) + (t & 3) * 64) * 16;
                if (planes(t)) {
                    note(put_block(ua, ub, R4, lane, r0, so + (slot * 8 + (wv - 1) * 2) * 4096, 4096), k);
                } else {
                    st128(u32x4{ua[0], ua[1], ua[2], ua[3]}, r0, doff, so, 17);
                    st128(u32x4{ub[0], ub[1], ub[2], ub[3]}, r0, doff + 256 * 16, so, 17);
                }
                wait_vm<18>();
                bar();
            };
            int t = j0, tn = j0 + 8;
            ld(va, t);
            setup(wsmp);
            asm volatile("s_waitcnt vmcnt(0)" : AGK_V16("+v", va) :: "memory");
            for (int i = 0;; i += 2) {
                step(va, vb, t, tn, i);
                if (tn >= NITEMS) break;
                t = tn; tn = __builtin_amdgcn_readfirstlane(lds_ld_asm(tk_a + ((i + 2) & 7) * 4));
                step(vb, va, t, tn, i + 1);
                if (tn >= NITEMS) break;
                t = tn; tn = __builtin_amdgcn_readfirstlane(lds_ld_asm(tk_a + ((i + 3) & 7) * 4));
            }
            asm volatile("s_waitcnt vmcnt(0)" : AGK_V16("+v", va) :: "memory");
            asm volatile("" : AGK_V16("+v", vb) :: "memory");
        }
        wait_vm<0>();
        bar();
        STAMPW(1, 1);
        return;
    }

    const int cid = bid - CB0;
    const int xg = cid & 7, xi = cid >> 3;
    const int rb = (xg & 1) * 4 + (xi & 3);
    const int cb = (xg >> 1) * 4 + (xi >> 2);
    volatile int* ctl = (volatile int*)(smem + 4 * 32768);
    const unsigned ctl_a = (unsigned)(uintptr_t)(__attribute__((address_space(3))) char*)(smem + 4 * 32768);
    if (tid < 84) lds_st(ctl + tid, tid < 16 ? 0 : -1);
    __syncthreads();
    const unsigned call = __builtin_amdgcn_readfirstlane(callv) + 1u;
    auto grp_src = [&](int g) { return g < 5 ? 2 * Q : g < 10 ? 2 * Q + 1 : 2 * ((Q + 1 + ((g - 10) >> 1)) & 3) + ((g - 10) & 1); };
    const int csrc = (cb >> 1) < 2 ? 8 + (cb >> 1) : 2 * ((Q + 1 + (((cb >> 1) - 2) >> 1)) & 3) + (((cb >> 1) - 2) & 1);

    if (wv >= 8) {
        const int psel = (lane >> 2) & 3;
        const int psrc = psel == 0 ? grp_src(2 * rb) : psel == 1 ? grp_src(2 * rb + 1) : csrc;
        const bool dup = psel == 3 || (psel == 1 && grp_src(2 * rb) == grp_src(2 * rb + 1));
        const char* pfb = me + IFLAG + (size_t)psrc * (NITEMS * 128) + (lane & 3) * 128;
        int cur = (lane >> 4) * NPOLL + (wv - 8);
        bool seen = dup;
        bool bad = false;
        unsigned long long t0 = __builtin_amdgcn_s_memrealtime();
        auto push = [&](int g) {
            const int st = __builtin_amdgcn_readlane(cur, 16 * g);
            int pos = 0;
            if (lane == 0) {
                pos = __hip_atomic_fetch_add((int*)ctl, 1, __ATOMIC_RELAXED, __HIP_MEMORY_SCOPE_WORKGROUP);
                lds_st(ctl + 16 + pos, st);
            }
            if ((lane >> 4) == g) { cur += 4 * NPOLL; seen = dup; }
        };
        while (__any(cur < NST)) {
            if (!seen && cur < NST) seen = ld_sys_asm(pfb + cur * (NSL * 128)) == call;
            const unsigned long long ok = __ballot(seen && cur < NST);
            bool any = false;
#pragma unroll
            for (int g = 0; g < 4; ++g)
                if (((unsigned)(ok >> (16 * g)) & 0xFFFFu) == 0xFFFFu) { push(g); any = true; }
            if (any) continue;
            if (give_up(t0, tmo, call)) {
                bad = true;
#pragma unroll
                for (int g = 0; g < 4; ++g)
                    while (__builtin_amdgcn_readlane(cur, 16 * g) < NST) push(g);
                break;
            }
            __builtin_amdgcn_s_sleep(1);
        }
        if (wv > 8) {
            if (bad && lane == 0) {
                __hip_atomic_store(tmo, call, __ATOMIC_RELAXED, __HIP_MEMORY_SCOPE_AGENT);
                lds_st(ctl + 1, 1);
            }
            asm volatile("s_waitcnt lgkmcnt(0)" ::: "memory");
            return;
        }
        const unsigned long long* gp = (const unsigned long long*)(me + GGRAN + ((call & 1u) * NR + (lane & 7)) * 128);
        t0 = __builtin_amdgcn_s_memrealtime();
        unsigned long long v;
        while (true) {
            v = __hip_atomic_load(gp, __ATOMIC_RELAXED, __HIP_MEMORY_SCOPE_SYSTEM);
            if (__all((unsigned)(v >> 32) == call)) break;
            if (give_up(t0, tmo, call)) { bad = true; break; }
            __builtin_amdgcn_s_sleep(1);
        }
        const unsigned am = wave_umax((unsigned)v);
        if (lane == 0) {
            if (bad) {
                __hip_atomic_store(tmo, call, __ATOMIC_RELAXED, __HIP_MEMORY_SCOPE_AGENT);
                lds_st(ctl + 1, 1);
            }
            lds_st(ctl + 12, (int)am);
        }
        asm volatile("s_waitcnt lgkmcnt(0)" ::: "memory");
        return;
    }

    if (wv >= 4) {
        const int lw = wv - 4;
        const char* xsrc = me + A_OFF + (size_t)(rb * 128 + lw * 32) * (BK * 2) + lane * 16;
        const char* wsrc = me + B_OFF + (size_t)(((cb >> 1) * 8 + lw * 2) * 256 + (cb & 1) * 128 + lane) * 16;
        int issued = 0, landed = 0;
        auto retire = [&]() {
            const int out = issued - landed;
            if (out == 1) wait_vm<0>(); else if (out == 2) wait_vm<8>(); else if (out == 3) wait_vm<16>(); else wait_vm<24>();
            ++landed;
            lds_st_asm(ctl_a + 16 + lw * 4, landed);
        };
        while (issued < NST) {
            u32x4 fw; unsigned long long qe;
            asm volatile("ds_read_b128 %0, %2 offset:32\n\tds_read2_b32 %1, %3 offset0:0 offset1:1\n\ts_waitcnt lgkmcnt(0)"
                         : "=&v"(fw), "=&v"(qe) : "v"(ctl_a), "v"(ctl_a + 64 + issued * 4) : "memory");
            const unsigned fa = fw.x < fw.y ? fw.x : fw.y, fb = fw.z < fw.w ? fw.z : fw.w;
            const int fr = __builtin_amdgcn_readfirstlane((int)(fa < fb ? fa : fb)) + 4;
            const int q0 = __builtin_amdgcn_readfirstlane((int)(unsigned)qe), q1 = __builtin_amdgcn_readfirstlane((int)(unsigned)(qe >> 32));
            const int rdy = q0 < 0 ? issued : q1 < 0 ? issued + 1 : issued + 2;
            const int lim = rdy < fr ? rdy : fr;
            bool progress = false;
#pragma unroll
            for (int k = 0; k < 2; ++k) {
                if (issued < lim) {
                    const int st = (k == 0 ? q0 : q1) & (NST - 1);
                    char* xb = smem + (issued & 3) * 32768 + lw * 4096;
                    char* wb = xb + 16384;
                    const char* xs = xsrc + (size_t)st * (AROWS * BK * 2);
                    const char* wsn = wsrc + (size_t)st * (8 * BCOLS * 16);
#pragma unroll
                    for (int i = 0; i < 4; ++i)
                        __builtin_amdgcn_global_load_lds((const unsigned*)(xs + i * 1024), (unsigned*)(xb + i * 1024), 16, 0, 17);
#pragma unroll
                    for (int i = 0; i < 4; ++i)
                        __builtin_amdgcn_global_load_lds((const unsigned*)(wsn + (i >> 1) * (256 * 16) + (i & 1) * 1024), (unsigned*)(wb + i * 1024), 16, 0, 17);
                    ++issued;
                    progress = true;
                }
                while (issued > landed && (issued - landed == 4 || vm_outstanding() <= 8 * (issued - landed - 1))) { retire(); progress = true; }
            }
            if (!progress) __builtin_amdgcn_s_sleep(1);
        }
        while (landed < NST) retire();
        return;
    }

    const int wm = wv >> 1, wn = wv & 1;
    const int ml = lane & 15, q = lane >> 4;
    f32x4 acc[4][4];
#pragma unroll
    for (int u = 0; u < 4; ++u)
#pragma unroll
        for (int t = 0; t < 4; ++t) acc[u][t] = f32x4{0.f, 0.f, 0.f, 0.f};
    const int xrd = (wm * 64 + ml) * 128;
    const int wrd = 16384 + (q * 128 + wn * 64 + ml) * 16;
    {
        const int gw = cid * 4 + wv;
        const char* xa = (const char*)p.x + (size_t)gw * (KDIM * 4) + lane * 16;
        const char* wa = (const char*)p.w + (size_t)gw * 1024 + lane * 16;
        u32x4 d0 = {0u, 0u, 0u, 0u}, d1 = {0u, 0u, 0u, 0u};
#pragma unroll
        for (int i = 0; i < 16; ++i)
            asm volatile("global_load_dwordx4 %0, %2, off\n\tglobal_load_dwordx4 %1, %3, off"
                         : "+v"(d0), "+v"(d1) : "v"(xa + i * 1024), "v"(wa + (size_t)i * (512 * 1024)) : "memory");
        asm volatile("s_waitcnt vmcnt(0)" : "+v"(d0), "+v"(d1) :: "memory");
    }
    STAMP(1);
    bf16x8 wf0[4], xf0[4], wf1[4], xf1[4];
#pragma unroll
    for (int t = 0; t < 4; ++t) { wf1[t] = bf16x8{}; xf1[t] = bf16x8{}; }
    const int xsw0 = (q ^ (ml & 7)) * 16, xsw1 = ((4 + q) ^ (ml & 7)) * 16;
    int full = 0;
    for (int s = 0; s < NST; ++s) {
        while (full <= s) {
            full = lds_min4_asm(ctl_a + 16);
            if (full <= s) __builtin_amdgcn_s_sleep(1);
        }
        const char* sb = smem + (s & 3) * 32768;
#pragma unroll
        for (int t = 0; t < 4; ++t) wf0[t] = *(const bf16x8*)(sb + wrd + t * 256);
#pragma unroll
        for (int u = 0; u < 4; ++u) xf0[u] = *(const bf16x8*)(sb + xrd + u * 2048 + xsw0);
        __builtin_amdgcn_sched_barrier(0);
#pragma unroll
        for (int u = 0; u < 4; ++u)
#pragma unroll
            for (int t = 0; t < 4; ++t)
                acc[u][t] = __builtin_amdgcn_mfma_f32_16x16x32_bf16(wf1[t], xf1[u], acc[u][t], 0, 0, 0);
        __builtin_amdgcn_sched_barrier(0);
        u32x4 ln;
        asm volatile("ds_read_b128 %0, %1" : "=&v"(ln) : "v"(ctl_a + 16) : "memory");
#pragma unroll
        for (int t = 0; t < 4; ++t) wf1[t] = *(const bf16x8*)(sb + wrd + (4 * 128 * 16) + t * 256);
#pragma unroll
        for (int u = 0; u < 4; ++u) xf1[u] = *(const bf16x8*)(sb + xrd + u * 2048 + xsw1);
        __builtin_amdgcn_sched_barrier(0);
#pragma unroll
        for (int u = 0; u < 2; ++u)
#pragma unroll
            for (int t = 0; t < 4; ++t)
                acc[u][t] = __builtin_amdgcn_mfma_f32_16x16x32_bf16(wf0[t], xf0[u], acc[u][t], 0, 0, 0);
        __builtin_amdgcn_sched_barrier(0);
        asm volatile("s_waitcnt lgkmcnt(0)" : "+v"(ln) :: "memory");
        lds_st_asm(ctl_a + 32 + wv * 4, s + 1);
        {
            const unsigned a = ln.x < ln.y ? ln.x : ln.y, b = ln.z < ln.w ? ln.z : ln.w;
            full = __builtin_amdgcn_readfirstlane((int)(a < b ? a : b));
        }
        __builtin_amdgcn_sched_barrier(0);
#pragma unroll
        for (int u = 2; u < 4; ++u)
#pragma unroll
            for (int t = 0; t < 4; ++t)
                acc[u][t] = __builtin_amdgcn_mfma_f32_16x16x32_bf16(wf0[t], xf0[u], acc[u][t], 0, 0, 0);
        __builtin_amdgcn_sched_barrier(0);
    }
#pragma unroll
    for (int u = 0; u < 4; ++u)
#pragma unroll
        for (int t = 0; t < 4; ++t)
            acc[u][t] = __builtin_amdgcn_mfma_f32_16x16x32_bf16(wf1[t], xf1[u], acc[u][t], 0, 0, 0);
    STAMP(3);

    float mx = 0.f;
#pragma unroll
    for (int u = 0; u < 4; ++u)
#pragma unroll
        for (int t = 0; t < 4; ++t)
            mx = fmaxf(fmaxf(mx, fmaxf(acc[u][t][0], acc[u][t][1])), fmaxf(acc[u][t][2], acc[u][t][3]));
    {
        const unsigned wmx = wave_umax(__float_as_uint(mx));
        int tk = 0;
        if (lane == 0) {
            __hip_atomic_fetch_max((unsigned*)(ctl + 2), wmx, __ATOMIC_RELAXED, __HIP_MEMORY_SCOPE_WORKGROUP);
            tk = __hip_atomic_fetch_add((int*)(ctl + 3), 1, __ATOMIC_RELAXED, __HIP_MEMORY_SCOPE_WORKGROUP);
        }
        tk = __builtin_amdgcn_readfirstlane(tk);
        if (tk == 3) {
            const unsigned tm = (unsigned)lds_ld(ctl + 2);
            if (lane == 0)
                __hip_atomic_store((unsigned long long*)(me + LGRAN + cid * 128), ((unsigned long long)call << 32) | tm,
                                   __ATOMIC_RELAXED, __HIP_MEMORY_SCOPE_AGENT);
        }
    }
    const int g = rb * 2 + wm;
    const int srk = grp_src(g);
    const int row0 = g < 5 ? 64 * g : g < 10 ? 64 * (g - 5) : 320 + 64 * ((2 - ((g - 10) >> 1)) & 3);
    const int owner = ((cb >> 1) < 2 ? rank : csrc) & 7;
    const int coff = (cb >> 1) < 2 ? cb * 128 : 256 * (rank & 1) + (cb & 1) * 128;
    const __amdgpu_buffer_rsrc_t ro = mk_rsrc(p.out[owner], (unsigned)(KDIM * NS));
    const unsigned ooff = (unsigned)((srk * MS + row0 + ml) * NS + coff + wn * 64 + q * 16);
    unsigned* dfl = (unsigned*)(p.ws[owner] + DFLAG + (rank * 16 + rb * 2 + (cb & 1)) * 128);
    bar();
    STAMP(4);
    mx = __int_as_float(lds_ld(ctl + 12));
    const bool bad = lds_ld(ctl + 1) != 0;
    const float inv = mx > 0.f ? 448.0f / mx : 0.f;
#pragma unroll
    for (int u = 0; u < 4; ++u) {
        u32x4 o;
#pragma unroll
        for (int t = 0; t < 4; ++t) {
            const float a0 = __builtin_amdgcn_fmed3f(acc[u][t][0] * inv, 0.f, 448.f), a1 = __builtin_amdgcn_fmed3f(acc[u][t][1] * inv, 0.f, 448.f);
            const float a2 = __builtin_amdgcn_fmed3f(acc[u][t][2] * inv, 0.f, 448.f), a3 = __builtin_amdgcn_fmed3f(acc[u][t][3] * inv, 0.f, 448.f);
            int r = __builtin_amdgcn_cvt_pk_fp8_f32(a0, a1, 0, false);
            r = __builtin_amdgcn_cvt_pk_fp8_f32(a2, a3, r, true);
            o[t] = bad ? 0x7F7F7F7Fu : (unsigned)r;
        }
        st128(o, ro, ooff, u * (16 * NS), 17);
    }
    wait_vm<0>();
    bar();
    if (tid == 0 && owner != rank) st_sys(dfl, call);
    STAMP(5);
}

}

extern "C" void kernel_launch(int rank, int nranks, ncclComm_t comm,
                              hipStream_t stream,
                              void* const* d_in, const int* in_sizes, int n_in,
                              void* d_out, int out_size,
                              const int* in_dtypes, int out_dtype,
                              void* const* d_out_all,
                              void* const* d_ws_all, size_t ws_bytes) {
    (void)comm; (void)in_sizes; (void)n_in; (void)d_out; (void)out_size; (void)in_dtypes; (void)out_dtype;
    if (nranks != agk::NR || (unsigned)rank >= (unsigned)agk::NR || ws_bytes < agk::WS_NEED) {
        fprintf(stderr, "unsupported configuration\n");
        return;
    }
    agk::Params p;
    p.x = (const float*)d_in[0];
    p.w = (const float*)d_in[1];
    for (int i = 0; i < agk::NR; ++i) {
        p.ws[i] = (char*)d_ws_all[i];
        p.out[i] = (unsigned char*)d_out_all[i];
    }
    p.rank = rank;
    hipLaunchKernelGGL(agk::ag_gemm_kernel, dim3(agk::GRID), dim3(agk::NTHR), 0, stream, p);
    HIP_CHECK(hipGetLastError());
}
```
